```python
import numpy as np

kernel_name = "gpu_ag_gemm_m4096_k4096_n16384_f32_relu"


ABSMAX_RTOL = 0.001

COMPUTE_DTYPE = "float32"
EPILOGUE = "relu"
IN_QUANT = None
OUT_QUANT = None
IN_DTYPES = {"x": "float32", "w_mat": "float32"}
WORLD_SIZE = 8

def setup_inputs():

    x = np.random.randn(4096, 4096).astype(np.float32)
    w_mat = (np.random.randn(4096, 16384) / np.sqrt(4096)).astype(np.float32)
    return {"x": x, "w_mat": w_mat}

def _epilogue(y):
    return np.maximum(y, 0.0).astype(np.float32)

def reference(x, w_mat):
    y = (x @ w_mat).astype(np.float32)
    return _epilogue(y)
```

```cpp
#include <hip/hip_runtime.h>
#include <rccl/rccl.h>
#include <cstdio>
#include <cstdint>

#define HIP_CHECK(expr) do { hipError_t _e = (expr); if (_e != hipSuccess) { \
    fprintf(stderr, "HIP error %s at %s:%d\n", hipGetErrorString(_e), __FILE__, __LINE__); \
    return; } } while (0)

namespace {

typedef _Float16 h16;
typedef _Float16 h16x8 __attribute__((ext_vector_type(8)));
typedef float f32x4 __attribute__((ext_vector_type(4)));
typedef unsigned u32x4 __attribute__((ext_vector_type(4)));
#define LAS __attribute__((address_space(3)))

#define S1 3
#define WIN 4

constexpr int NR = 8;
constexpr int GM = 4096;
constexpr int GK = 4096;
constexpr int NL = 2048;
constexpr int MS = GM / NR;
constexpr int NKB = GK / 32;
constexpr int NRG = GM / 16;
constexpr int NCG = NL / 16;
constexpr int RG_PER = MS / 16;

constexpr int UNITS = RG_PER;
__host__ __device__ constexpr bool slab_start(int kb) { return kb < 112 ? (kb & 7) == 0 : (kb == 112 || kb == 118 || kb == 122 || kb == 126); }
constexpr int NCTR = NKB / 2;

constexpr int PUSH_BLOCKS = RG_PER;
constexpr int WCONV_BLOCKS = NKB;
constexpr int GEMM_BLOCKS = 224;

constexpr size_t OFF_CTR = 0;
constexpr size_t OFF_TMO = 1024;
constexpr size_t OFF_WCNT = 2048;
constexpr size_t OFF_WC2 = 96 << 10;
constexpr size_t OFF_PMAX = 256 << 10;
constexpr size_t OFF_ARR = 4096;
constexpr size_t OFF_XS  = 128 << 10;
constexpr size_t OFF_WSC = 192 << 10;
constexpr size_t OFF_XH  = 1 << 20;
constexpr size_t XH_BYTES = (size_t)GM * GK * 2;
constexpr size_t OFF_WT  = OFF_XH + 2 * XH_BYTES;
constexpr size_t WS_NEED = OFF_WT + (size_t)NL * GK * 2;

constexpr unsigned long long DEADLINE_TICKS = 50000000ull;

constexpr int NS = 4;
constexpr int STG = 28 * 1024;
constexpr int LDS_BYTES = 131072 + 4096;

struct Params {
    const float* x;
    const float* w;
    float* out;
    char* ws[NR];
    int rank;
    int phase;
};

__device__ __forceinline__ unsigned draw_call(unsigned* ctr, LAS unsigned* slot) {
    if (threadIdx.x == 0) {
        unsigned t = __hip_atomic_fetch_add(ctr, 1u, __ATOMIC_RELAXED, __HIP_MEMORY_SCOPE_AGENT);
        *slot = t / gridDim.x + 1u;
    }
    __syncthreads();
    unsigned c = __builtin_amdgcn_readfirstlane(*slot);
    __syncthreads();
    return c;
}

__device__ __forceinline__ void scale_from_amax(float amax, float& mult, float& inv) {
    int ef = (int)((__float_as_uint(amax) >> 23) & 0xffu);
    int e = ef - 127 - 14;
    if (ef == 0xff || amax == 0.f) e = 0;
    e = e < -100 ? -100 : (e > 113 ? 113 : e);
    mult = __uint_as_float((unsigned)(127 - e) << 23);
    inv  = __uint_as_float((unsigned)(127 + e) << 23);
}

__device__ __forceinline__ float amax4(f32x4 v) {
    return fmaxf(fmaxf(fabsf(v[0]), fabsf(v[1])), fmaxf(fabsf(v[2]), fabsf(v[3])));
}

#define WAIT_V(n) asm volatile("s_waitcnt vmcnt(" #n ")" ::: "memory")
#define WAIT_VN(n) asm volatile("s_waitcnt vmcnt(%0)" :: "n"(n) : "memory")

__device__ __forceinline__ void role_push(const Params& p, LAS char* lds, int rg, unsigned call, bool local) {
    const int tid = threadIdx.x, wid = __builtin_amdgcn_readfirstlane(tid >> 6), lane = tid & 63;
    const int par = call & 1;
    const float* xb = p.x + (size_t)rg * 16 * GK;
    LAS float* rinv = (LAS float*)(lds + 131072 + 64);
#pragma unroll
    for (int r = 0; r < 2; ++r) {
        const int row = wid * 2 + r;
        const float* src = xb + (size_t)row * GK + lane * 4;
        f32x4 v[16];
#pragma unroll
        for (int c = 0; c < 16; ++c) v[c] = *(const f32x4*)(src + c * 256);
        float am = 0.f;
#pragma unroll
        for (int c = 0; c < 16; ++c) am = fmaxf(am, amax4(v[c]));
#pragma unroll
        for (int o = 32; o >= 1; o >>= 1) am = fmaxf(am, __shfl_xor(am, o));
        float mult, inv;
        scale_from_amax(am, mult, inv);
        if (lane == 0) rinv[row] = inv;
        typedef _Float16 h16x4 __attribute__((ext_vector_type(4)));
#pragma unroll
        for (int c = 0; c < 16; ++c) {
            const int k = c * 256 + lane * 4;
            h16x4 h;
#pragma unroll
            for (int j = 0; j < 4; ++j) h[j] = (h16)(v[c][j] * mult);
            *(LAS h16x4*)(lds + (k >> 5) * 1024 + (((k >> 3) & 3) * 16 + row) * 16 + (k & 4) * 2) = h;
        }
    }
    __syncthreads();
    const int grg = p.rank * RG_PER + rg;
    if (local) {
        char* myws = p.ws[p.rank];
        __amdgpu_buffer_rsrc_t rs = __builtin_amdgcn_make_buffer_rsrc(
            (void*)(myws + OFF_XH + (size_t)par * XH_BYTES), 0, (int)XH_BYTES, 0x00020000);
        if (tid < 16) __hip_atomic_store((float*)(myws + OFF_XS) + par * GM + grg * 16 + tid, rinv[tid], __ATOMIC_RELAXED, __HIP_MEMORY_SCOPE_AGENT);
#pragma unroll
        for (int i = 0; i < 16; ++i) {
            const int kb = i * 8 + wid;
            u32x4 dat = *(const LAS u32x4*)(lds + kb * 1024 + lane * 16);
            __builtin_amdgcn_raw_buffer_store_b128(dat, rs, (kb * NRG + grg) * 1024 + lane * 16, 0, 16  );
        }
        WAIT_V(0);
        __syncthreads();
        if (tid == 0) __hip_atomic_fetch_add((unsigned*)(myws + OFF_ARR + (size_t)(p.rank * NCTR) * 128), 1u, __ATOMIC_RELAXED, __HIP_MEMORY_SCOPE_AGENT);
        return;
    }
    if (wid == 7) return;
    const int d = (p.rank + 1 + wid) & 7;
    char* dws = p.ws[d];
    __amdgpu_buffer_rsrc_t rs = __builtin_amdgcn_make_buffer_rsrc(
        (void*)(dws + OFF_XH + (size_t)par * XH_BYTES), 0, (int)XH_BYTES, 0x00020000);
    unsigned* arr = (unsigned*)(dws + OFF_ARR + (size_t)(p.rank * NCTR) * 128);
    if (lane < 16) {
        float* xs = (float*)(dws + OFF_XS) + par * GM + grg * 16 + lane;
        __hip_atomic_store(xs, rinv[lane], __ATOMIC_RELAXED, __HIP_MEMORY_SCOPE_SYSTEM);
    }
    int cur = 0;
    for (int b8 = 0; b8 < NKB; b8 += 8) {
#pragma unroll
        for (int i = 0; i < 8; ++i) {
            const int kb = b8 + i;
            u32x4 dat = *(const LAS u32x4*)(lds + kb * 1024 + lane * 16);
            WAIT_VN(WIN - 1);
            __builtin_amdgcn_raw_buffer_store_b128(dat, rs, (kb * NRG + grg) * 1024 + lane * 16, 0, 17  );
            if (kb + 1 == NKB || slab_start(kb + 1)) {
                WAIT_V(0);
                if (lane == 0) __hip_atomic_fetch_add(arr + (cur >> 1) * 32, 1u, __ATOMIC_RELAXED, __HIP_MEMORY_SCOPE_SYSTEM);
                cur = kb + 1;
            }
        }
    }
}

__device__ __forceinline__ void poll(const unsigned* ctr, unsigned target, unsigned* tmo, unsigned call, int lane);
__device__ __forceinline__ void role_wconv(const Params& p, LAS char* lds, int g, unsigned call) {
    const int tid = threadIdx.x, wid = __builtin_amdgcn_readfirstlane(tid >> 6), lane = tid & 63;
    char* myws = p.ws[p.rank];
    const int sl = g >> 5, cq = lane & 15, col0 = (g & 31) * 64 + cq * 4;
    const int kbl = wid * 4 + (lane >> 4), kb = sl * 32 + kbl;
    const float* src = p.w + (size_t)(kb * 32) * NL + col0;
    f32x4 v[32];
#pragma unroll
    for (int r = 0; r < 32; ++r) v[r] = *(const f32x4*)(src + (size_t)r * NL);
    f32x4 am = {0.f, 0.f, 0.f, 0.f};
#pragma unroll
    for (int r = 0; r < 32; ++r)
#pragma unroll
        for (int j = 0; j < 4; ++j) am[j] = fmaxf(am[j], fabsf(v[r][j]));
#pragma unroll
    for (int j = 0; j < 4; ++j) { am[j] = fmaxf(am[j], __shfl_xor(am[j], 16)); am[j] = fmaxf(am[j], __shfl_xor(am[j], 32)); }
    LAS f32x4* red = (LAS f32x4*)(lds + 131072 + 1024);
    if (lane < 16) red[wid * 16 + lane] = am;
    __syncthreads();
#pragma unroll
    for (int w = 0; w < 8; ++w) { f32x4 o = red[w * 16 + cq];
#pragma unroll
        for (int j = 0; j < 4; ++j) am[j] = fmaxf(am[j], o[j]); }
    float* pmax = (float*)(myws + OFF_PMAX);
    if (wid == 0) {
        if (lane < 16) {
#pragma unroll
            for (int j = 0; j < 4; ++j) __hip_atomic_store(pmax + sl * NL + col0 + j, am[j], __ATOMIC_RELAXED, __HIP_MEMORY_SCOPE_AGENT);
        }
        WAIT_V(0);
        unsigned* c2 = (unsigned*)(myws + OFF_WC2 + (g & 31) * 128);
        if (lane == 0) __hip_atomic_fetch_add(c2, 1u, __ATOMIC_RELAXED, __HIP_MEMORY_SCOPE_AGENT);
        poll(c2, call * 4u, (unsigned*)(myws + OFF_TMO), call, lane);
    }
    __syncthreads();
#pragma unroll
    for (int q4 = 0; q4 < 4; ++q4)
#pragma unroll
        for (int j = 0; j < 4; ++j) am[j] = fmaxf(am[j], __hip_atomic_load(pmax + q4 * NL + col0 + j, __ATOMIC_RELAXED, __HIP_MEMORY_SCOPE_AGENT));
    float mult[4], inv[4];
#pragma unroll
    for (int j = 0; j < 4; ++j) scale_from_amax(am[j], mult[j], inv[j]);
    if (sl == 0 && tid < 16) {
#pragma unroll
        for (int j = 0; j < 4; ++j) __hip_atomic_store((float*)(myws + OFF_WSC) + col0 + j, inv[j], __ATOMIC_RELAXED, __HIP_MEMORY_SCOPE_AGENT);
    }
    LAS char* fb = lds + wid * 16384;
    const int cgl = cq >> 2, c0 = (cq & 3) * 4;
#pragma unroll
    for (int kq = 0; kq < 4; ++kq)
#pragma unroll
        for (int j = 0; j < 4; ++j) {
            h16x8 h;
#pragma unroll
            for (int r = 0; r < 8; ++r) h[r] = (h16)(v[kq * 8 + r][j] * mult[j]);
            *(LAS h16x8*)(fb + ((lane >> 4) * 4 + cgl) * 1024 + (kq * 16 + c0 + j) * 16) = h;
        }
    asm volatile("s_waitcnt lgkmcnt(0)" ::: "memory");
    __amdgpu_buffer_rsrc_t rs = __builtin_amdgcn_make_buffer_rsrc((void*)(myws + OFF_WT), 0, NL * GK * 2, 0x00020000);
#pragma unroll
    for (int f = 0; f < 16; ++f) {
        const u32x4 dat = *(const LAS u32x4*)(fb + f * 1024 + lane * 16);
        const int kbf = sl * 32 + wid * 4 + (f >> 2), cg = (g & 31) * 4 + (f & 3);
        __builtin_amdgcn_raw_buffer_store_b128(dat, rs, (kbf * NCG + cg) * 1024 + lane * 16, 0, 16  );
    }
    WAIT_V(0);
    __syncthreads();
    if (tid == 0) __hip_atomic_fetch_add((unsigned*)(myws + OFF_WCNT), 1u, __ATOMIC_RELAXED, __HIP_MEMORY_SCOPE_AGENT);
}

__device__ __forceinline__ void poll(const unsigned* ctr, unsigned target, unsigned* tmo, unsigned call, int lane) {
    const unsigned long long t0 = __builtin_amdgcn_s_memrealtime();
    while ((int)((unsigned)__builtin_amdgcn_readfirstlane(__hip_atomic_load(ctr, __ATOMIC_RELAXED, __HIP_MEMORY_SCOPE_SYSTEM)) - target) < 0) {
        if ((unsigned)__builtin_amdgcn_readfirstlane(__hip_atomic_load(tmo, __ATOMIC_RELAXED, __HIP_MEMORY_SCOPE_AGENT)) == call
            || __builtin_amdgcn_s_memrealtime() - t0 > DEADLINE_TICKS) {
            if (lane == 0) __hip_atomic_store(tmo, call, __ATOMIC_RELAXED, __HIP_MEMORY_SCOPE_AGENT);
            break;
        }
        __builtin_amdgcn_s_sleep(2);
    }
}

__device__ __forceinline__ void role_gemm(const Params& p, LAS char* lds, int bid, unsigned call) {
    const int tid = threadIdx.x, wid = __builtin_amdgcn_readfirstlane(tid >> 6), lane = tid & 63;
    const int wr = wid >> 1, wc = wid & 1;
    char* myws = p.ws[p.rank];
    const int par = call & 1;
    unsigned* tmo = (unsigned*)(myws + OFF_TMO);

    const int xcd = bid & 7, sl = bid >> 3;
    const int band = 7 * (xcd >> 2) + (sl % 7), pn = 4 * (xcd & 3) + (sl / 7);
    const int rg0 = band * 18 + (band < 4 ? band : 4);
    const int nrg = band < 4 ? 19 : 18;
    const int srcA = rg0 / RG_PER, srcB = (rg0 + nrg - 1) / RG_PER;
    const bool needA = srcA != p.rank, needB = (srcB != srcA) && (srcB != p.rank);
    const bool needL = (srcA == p.rank) || (srcB == p.rank);
    const unsigned* arrA = (const unsigned*)(myws + OFF_ARR + (size_t)(srcA * NCTR) * 128);
    const unsigned* arrB = (const unsigned*)(myws + OFF_ARR + (size_t)(srcB * NCTR) * 128);
    const unsigned target = call * UNITS;

    const char* gsrc[4];
    int fidx[4];
#pragma unroll
    for (int i = 0; i < 4; ++i) {
        const int f = (i < 3) ? wid * 3 + i : 24 + wid;
        fidx[i] = f;
        const int ra = rg0 + (f < nrg ? f : nrg - 1);
        gsrc[i] = (f < 20) ? myws + OFF_XH + (size_t)par * XH_BYTES + (size_t)ra * 1024 + lane * 16
                           : myws + OFF_WT + (size_t)(pn * 8 + ((f - 20) & 7)) * 1024 + lane * 16;
    }
#define STAGE(kb, slot) do { _Pragma("unroll") for (int _i = 0; _i < 4; ++_i) { if (_i < 3 || wid < 4) \
        __builtin_amdgcn_global_load_lds((const unsigned*)(gsrc[_i] + (size_t)(kb) * ((fidx[_i] < 20) ? NRG * 1024 : NCG * 1024)), \
            (LAS unsigned*)(lds + (slot) * STG + fidx[_i] * 1024), 16, 0, 0); } } while (0)
#define BAR __builtin_amdgcn_s_barrier()
#define WAIT_STAGES(n) do { if (wid < 4) WAIT_VN((n) * 4); else WAIT_VN((n) * 3); } while (0)
    f32x4 acc[5][4];
#pragma unroll
    for (int m = 0; m < 5; ++m)
#pragma unroll
        for (int n = 0; n < 4; ++n) acc[m][n] = (f32x4){0.f, 0.f, 0.f, 0.f};
    h16x8 A0[5], B0[4], A1[5], B1[4];
    const int aoff = (wr * 5) * 1024 + lane * 16, boff = (20 + wc * 4) * 1024 + lane * 16;
#define LOADF(Af, Bf, slot) do { const LAS char* _b = lds + (slot) * STG; \
        _Pragma("unroll") for (int n = 0; n < 4; ++n) Bf[n] = *(const LAS h16x8*)(_b + boff + n * 1024); \
        _Pragma("unroll") for (int m = 0; m < 5; ++m) Af[m] = *(const LAS h16x8*)(_b + aoff + m * 1024); } while (0)
#define MMA(Af, Bf, m0, m1) do { _Pragma("unroll") for (int m = m0; m < m1; ++m) _Pragma("unroll") for (int n = 0; n < 4; ++n) \
        acc[m][n] = __builtin_amdgcn_mfma_f32_16x16x32_f16(Bf[n], Af[m], acc[m][n], 0, 0, 0); } while (0)
#define SCHEDULE do { _Pragma("unroll") for (int _q = 0; _q < 9; ++_q) { __builtin_amdgcn_sched_group_barrier(0x8, 1, 0); __builtin_amdgcn_sched_group_barrier(0x100, 1, 0); } \
        __builtin_amdgcn_sched_group_barrier(0x8, 3, 0); } while (0)
#define GATE(kb) do { if (needA) poll(arrA + ((kb) >> 1) * 32, target, tmo, call, lane); if (needB) poll(arrB + ((kb) >> 1) * 32, target, tmo, call, lane); } while (0)

    if (wid == 0) {
        poll((const unsigned*)(myws + OFF_WCNT), call * (unsigned)WCONV_BLOCKS, tmo, call, lane);
        if (needL) poll((const unsigned*)(myws + OFF_ARR + (size_t)(p.rank * NCTR) * 128), target, tmo, call, lane);
        GATE(0);
        __builtin_amdgcn_fence(__ATOMIC_ACQUIRE, "");
        WAIT_V(0);
    }
    asm volatile("" ::: "memory"); BAR; asm volatile("" ::: "memory");
    const int fr = lane & 15, fq = lane >> 4;
    f32x4 cs[4];
    float rsc[5];
    {
        const float* xs = (const float*)(myws + OFF_XS) + par * GM;
        const float* wsc = (const float*)(myws + OFF_WSC) + pn * 128 + wc * 64 + 4 * fq;
#pragma unroll
        for (int n = 0; n < 4; ++n)
#pragma unroll
            for (int jj = 0; jj < 4; ++jj) cs[n][jj] = __hip_atomic_load(wsc + n * 16 + jj, __ATOMIC_RELAXED, __HIP_MEMORY_SCOPE_AGENT);
#pragma unroll
        for (int m = 0; m < 5; ++m) {
            const int f = wr * 5 + m;
            rsc[m] = __hip_atomic_load(xs + (rg0 + (f < nrg ? f : nrg - 1)) * 16 + fr, __ATOMIC_RELAXED, __HIP_MEMORY_SCOPE_SYSTEM);
        }
        WAIT_V(0);
    }
    STAGE(0, 0); STAGE(1, 1); STAGE(2, 2); STAGE(3, 3);
    WAIT_STAGES(3); BAR;
    LOADF(A0, B0, 0);
    WAIT_STAGES(2);
    asm volatile("s_waitcnt lgkmcnt(0)" ::: "memory");
    BAR;
    for (int j = 0; j < NKB; j += 2) {
        LOADF(A1, B1, (j + 1) & 3);
        MMA(A0, B0, 0, 3);
        SCHEDULE;
        { const int kn = (j + 4 < NKB) ? j + 4 : 0; STAGE(kn, j & 3); }
        MMA(A0, B0, 3, 5);
        WAIT_STAGES(2);
        asm volatile("s_waitcnt lgkmcnt(0)" ::: "memory");
        BAR;
        LOADF(A0, B0, (j + 2) & 3);
        MMA(A1, B1, 0, 3);
        SCHEDULE;
        { const int kn = (j + 5 < NKB) ? j + 5 : 0; STAGE(kn, (j + 1) & 3); }
        MMA(A1, B1, 3, 5);
        if (wid == 0 && j + 6 < NKB && slab_start(j + 6)) {
            GATE(j + 6);
        }
        WAIT_STAGES(2);
        asm volatile("s_waitcnt lgkmcnt(0)" ::: "memory");
        BAR;
    }
    WAIT_V(0);
    BAR;
    const bool bad = __hip_atomic_load(tmo, __ATOMIC_RELAXED, __HIP_MEMORY_SCOPE_AGENT) == call;
    const float qnan = __builtin_nanf("");
    LAS char* eb = lds + wid * 8704;
#pragma unroll
    for (int m = 0; m < 5; ++m) {
        const int f = wr * 5 + m;
        if (f < nrg) {
            LAS char* b = eb + (m & 1) * 4352;
#pragma unroll
            for (int n = 0; n < 4; ++n) {
                f32x4 o = (acc[m][n] * rsc[m]) * cs[n];
#pragma unroll
                for (int jj = 0; jj < 4; ++jj) o[jj] = bad ? qnan : fmaxf(o[jj], 0.f);
                *(LAS f32x4*)(b + fr * 272 + n * 64 + fq * 16) = o;
            }
            asm volatile("s_waitcnt lgkmcnt(0)" ::: "memory");
            float* op = p.out + (size_t)((rg0 + f) * 16 + fq) * NL + pn * 128 + wc * 64 + fr * 4;
#pragma unroll
            for (int i = 0; i < 4; ++i) {
                const f32x4 o = *(const LAS f32x4*)(b + (i * 4 + fq) * 272 + fr * 16);
                *(f32x4*)(op + (size_t)(i * 4) * NL) = o;
            }
        }
    }
#undef STAGE
#undef BAR
#undef LOADF
#undef MMA
#undef GATE
#undef SCHEDULE
#undef WAIT_STAGES
}

__global__ void __launch_bounds__(512, 2) ag_gemm_kernel(Params p) {
    extern __shared__ __attribute__((aligned(16))) char smem[];
    LAS char* lds = (LAS char*)smem;
    const unsigned call = draw_call((unsigned*)(p.ws[p.rank] + OFF_CTR + p.phase * 128), (LAS unsigned*)(lds + 131072));
    const int b = blockIdx.x;
    int role;
    if (b < PUSH_BLOCKS) { role = 3; role_push(p, lds, b, call, false); }
    else {
        role = 2;
        const int g = b - PUSH_BLOCKS;
        if (g < WCONV_BLOCKS) role_wconv(p, lds, g, call);
        else if (g < WCONV_BLOCKS + RG_PER) role_push(p, lds, g - WCONV_BLOCKS, call, true);
        role_gemm(p, lds, g, call);
    }
}

}

extern "C" void kernel_launch(int rank, int nranks, ncclComm_t comm,
                              hipStream_t stream,
                              void* const* d_in, const int* in_sizes, int n_in,
                              void* d_out, int out_size,
                              const int* in_dtypes, int out_dtype,
                              void* const* d_out_all,
                              void* const* d_ws_all, size_t ws_bytes) {
    static bool init = false;
    if (nranks != NR || ws_bytes < WS_NEED) { fprintf(stderr, "unexpected configuration\n"); return; }
    if (!init) {
        HIP_CHECK(hipFuncSetAttribute((const void*)ag_gemm_kernel, hipFuncAttributeMaxDynamicSharedMemorySize, LDS_BYTES));
        init = true;
    }
    Params p;
    p.x = (const float*)d_in[0];
    p.w = (const float*)d_in[1];
    p.out = (float*)d_out;
    for (int i = 0; i < NR; ++i) p.ws[i] = (char*)d_ws_all[i];
    p.rank = rank;
    p.phase = 0;
    ag_gemm_kernel<<<GEMM_BLOCKS + PUSH_BLOCKS, 512, LDS_BYTES, stream>>>(p);
    HIP_CHECK(hipGetLastError());
}
```

```cpp
#include <hip/hip_runtime.h>
#include <rccl/rccl.h>
#include <cstdio>
#include <cstdint>

#define HIP_CHECK(expr) do { hipError_t _e = (expr); if (_e != hipSuccess) { \
    fprintf(stderr, "HIP error %s at %s:%d\n", hipGetErrorString(_e), __FILE__, __LINE__); \
    return; } } while (0)

namespace {

typedef _Float16 h16;
typedef _Float16 h16x8 __attribute__((ext_vector_type(8)));
typedef float f32x4 __attribute__((ext_vector_type(4)));
typedef unsigned u32x4 __attribute__((ext_vector_type(4)));
#define LAS __attribute__((address_space(3)))

#define S1 3
#define WDELAY 0
#define OUTNT 1
#define LOCK 0
#define WIN 4

constexpr int NR = 8;
constexpr int GM = 4096;
constexpr int GK = 4096;
constexpr int NL = 2048;
constexpr int MS = GM / NR;
constexpr int NKB = GK / 32;
constexpr int NRG = GM / 16;
constexpr int NCG = NL / 16;
constexpr int RG_PER = MS / 16;

constexpr int UNITS = RG_PER;
__host__ __device__ constexpr bool slab_start(int kb) { return kb < 64 ? (kb & 7) == 0 : (kb < 112 ? (kb & 3) == 0 : (kb & 1) == 0); }
constexpr int NSLAB = 28;
__host__ __device__ constexpr int slab_first(int s) { return s < 8 ? s * 8 : (s < 20 ? 64 + (s - 8) * 4 : 112 + (s - 20) * 2); }
__host__ __device__ constexpr int slab_idx(int kb) { return kb < 64 ? kb >> 3 : (kb < 112 ? 8 + ((kb - 64) >> 2) : 20 + ((kb - 112) >> 1)); }

constexpr int PUSH_BLOCKS = RG_PER;
constexpr int WCONV_BLOCKS = NKB;
constexpr int GEMM_BLOCKS = 224;

constexpr size_t OFF_CTR = 0;
constexpr size_t OFF_TMO = 1024;
constexpr size_t OFF_WCNT = 2048;
constexpr size_t OFF_LP = 8192;
constexpr size_t OFF_WC2 = 96 << 10;
constexpr size_t OFF_PMAX = 256 << 10;
constexpr size_t OFF_ARR = 4096;
constexpr size_t OFF_XS  = 128 << 10;
constexpr size_t OFF_XS0 = 160 << 10;
constexpr size_t OFF_WSC = 192 << 10;
constexpr size_t OFF_XH  = 1 << 20;
constexpr size_t XH_BYTES = (size_t)GM * GK * 2;
constexpr size_t OFF_WT  = OFF_XH + 2 * XH_BYTES;
constexpr size_t WS_NEED = OFF_WT + (size_t)NL * GK * 2;

constexpr unsigned long long DEADLINE_TICKS = 50000000ull;

#define NSTAGES 4
constexpr int NS = NSTAGES;
constexpr int STG = 28 * 1024;
constexpr int CTL = (NS * STG > 131072) ? NS * STG : 131072;
constexpr int LDS_BYTES = CTL + 4096;

struct Params {
    const float* x;
    const float* w;
    float* out;
    char* ws[NR];
    int rank;
    int phase;
};

__device__ __forceinline__ unsigned draw_call(unsigned* ctr, LAS unsigned* slot) {
    if (threadIdx.x == 0) {
        unsigned t = __hip_atomic_fetch_add(ctr, 1u, __ATOMIC_RELAXED, __HIP_MEMORY_SCOPE_AGENT);
        *slot = t / gridDim.x + 1u;
    }
    __syncthreads();
    unsigned c = __builtin_amdgcn_readfirstlane(*slot);
    __syncthreads();
    return c;
}

__device__ __forceinline__ void scale_from_amax(float amax, float& mult, float& inv) {
    int ef = (int)((__float_as_uint(amax) >> 23) & 0xffu);
    int e = ef - 127 - 14;
    if (ef == 0xff || amax == 0.f) e = 0;
    e = e < -100 ? -100 : (e > 113 ? 113 : e);
    mult = __uint_as_float((unsigned)(127 - e) << 23);
    inv  = __uint_as_float((unsigned)(127 + e) << 23);
}

__device__ __forceinline__ float amax4(f32x4 v) {
    return fmaxf(fmaxf(fabsf(v[0]), fabsf(v[1])), fmaxf(fabsf(v[2]), fabsf(v[3])));
}

#define WAIT_V(n) asm volatile("s_waitcnt vmcnt(" #n ")" ::: "memory")
#define WAIT_VN(n) asm volatile("s_waitcnt vmcnt(%0)" :: "n"(n) : "memory")

__device__ __forceinline__ void poll_agent(const unsigned* ctr, unsigned target, unsigned* tmo, unsigned call, int lane) {
    const unsigned long long t0 = __builtin_amdgcn_s_memrealtime();
    while ((int)((unsigned)__builtin_amdgcn_readfirstlane(__hip_atomic_load(ctr, __ATOMIC_RELAXED, __HIP_MEMORY_SCOPE_AGENT)) - target) < 0) {
        if ((unsigned)__builtin_amdgcn_readfirstlane(__hip_atomic_load(tmo, __ATOMIC_RELAXED, __HIP_MEMORY_SCOPE_AGENT)) == call
            || __builtin_amdgcn_s_memrealtime() - t0 > DEADLINE_TICKS) {
            if (lane == 0) __hip_atomic_store(tmo, call, __ATOMIC_RELAXED, __HIP_MEMORY_SCOPE_AGENT);
            break;
        }
        __builtin_amdgcn_s_sleep(1);
    }
}

__device__ __forceinline__ unsigned role_push(const Params& p, LAS char* lds, int rg, bool local) {
    typedef _Float16 h16x4 __attribute__((ext_vector_type(4)));
    const int tid = threadIdx.x, wid = __builtin_amdgcn_readfirstlane(tid >> 6), lane = tid & 63;
    unsigned ticket = 0;
    if (tid == 0) ticket = __hip_atomic_fetch_add((unsigned*)(p.ws[p.rank] + OFF_CTR), 1u, __ATOMIC_RELAXED, __HIP_MEMORY_SCOPE_AGENT);
    const float* xb = p.x + (size_t)rg * 16 * GK;
    LAS float* rinv = (LAS float*)(lds + CTL + 64);
    f32x4 v[2][16];
#pragma unroll
    for (int c = 0; c < 16; ++c)
#pragma unroll
        for (int r = 0; r < 2; ++r) v[r][c] = *(const f32x4*)(xb + (size_t)(wid * 2 + r) * GK + lane * 4 + c * 256);
    const int kl = lane * 4;
    const int loff = (kl >> 5) * 1024 + (((kl >> 3) & 3) * 16) * 16 + (kl & 4) * 2;
#pragma unroll
    for (int r = 0; r < 2; ++r) {
        const int row = wid * 2 + r;
        float am = amax4(v[r][0]);
#pragma unroll
        for (int o = 32; o >= 1; o >>= 1) am = fmaxf(am, __shfl_xor(am, o));
        float mult, inv;
        scale_from_amax(am, mult, inv);
        if (lane == 0) rinv[16 + row] = inv;
        h16x4 h;
#pragma unroll
        for (int j = 0; j < 4; ++j) h[j] = (h16)(v[r][0][j] * mult);
        *(LAS h16x4*)(lds + loff + row * 16) = h;
    }
    if (tid == 0) *(LAS unsigned*)(lds + CTL) = ticket / gridDim.x + 1u;
    __syncthreads();
    const unsigned call = __builtin_amdgcn_readfirstlane(*(LAS unsigned*)(lds + CTL));
    const int par = call & 1;
    const int grg = p.rank * RG_PER + rg;
    const bool sender = wid < 7;
    char* dws = p.ws[(p.rank + 1 + wid) & 7];
    __amdgpu_buffer_rsrc_t rs = __builtin_amdgcn_make_buffer_rsrc(
        (void*)(dws + OFF_XH + (size_t)par * XH_BYTES), 0, (int)XH_BYTES, 0x00020000);
    unsigned* arr = (unsigned*)(dws + OFF_ARR + (size_t)p.rank * 256);
    if (local && sender) {
        if (lane < 16) __hip_atomic_store((float*)(dws + OFF_XS0) + par * GM + grg * 16 + lane, rinv[16 + lane], __ATOMIC_RELAXED, __HIP_MEMORY_SCOPE_SYSTEM);
#pragma unroll
        for (int kb = 0; kb < 8; ++kb) {
            u32x4 dat = *(const LAS u32x4*)(lds + kb * 1024 + lane * 16);
            __builtin_amdgcn_raw_buffer_store_b128(dat, rs, (kb * NRG + grg) * 1024 + lane * 16, 0, 17  );
        }
    }
#pragma unroll
    for (int r = 0; r < 2; ++r) {
        const int row = wid * 2 + r;
        float am = 0.f;
#pragma unroll
        for (int c = 0; c < 16; ++c) am = fmaxf(am, amax4(v[r][c]));
#pragma unroll
        for (int o = 32; o >= 1; o >>= 1) am = fmaxf(am, __shfl_xor(am, o));
        float mult, inv;
        scale_from_amax(am, mult, inv);
        if (lane == 0) rinv[row] = inv;
#pragma unroll
        for (int c = 1; c < 16; ++c) {
            h16x4 h;
#pragma unroll
            for (int j = 0; j < 4; ++j) h[j] = (h16)(v[r][c][j] * mult);
            *(LAS h16x4*)(lds + c * 8192 + loff + row * 16) = h;
        }
    }
    if (local && sender) {
        WAIT_V(0);
        if (lane == 0) __hip_atomic_fetch_add(arr, 1u, __ATOMIC_RELAXED, __HIP_MEMORY_SCOPE_SYSTEM);
    }
    __syncthreads();
    if (local) {
        dws = p.ws[p.rank];
        rs = __builtin_amdgcn_make_buffer_rsrc((void*)(dws + OFF_XH + (size_t)par * XH_BYTES), 0, (int)XH_BYTES, 0x00020000);
        if (tid < 16) __hip_atomic_store((float*)(dws + OFF_XS) + par * GM + grg * 16 + tid, rinv[tid], __ATOMIC_RELAXED, __HIP_MEMORY_SCOPE_AGENT);
        else if (tid < 32) __hip_atomic_store((float*)(dws + OFF_XS0) + par * GM + grg * 16 + (tid - 16), rinv[tid], __ATOMIC_RELAXED, __HIP_MEMORY_SCOPE_AGENT);
#pragma unroll
        for (int i = 0; i < 16; ++i) {
            const int kb = i * 8 + wid;
            u32x4 dat = *(const LAS u32x4*)(lds + kb * 1024 + lane * 16);
            __builtin_amdgcn_raw_buffer_store_b128(dat, rs, (kb * NRG + grg) * 1024 + lane * 16, 0, 16  );
        }
        WAIT_V(0);
        __syncthreads();
        if (tid == 0) __hip_atomic_fetch_add((unsigned*)(dws + OFF_ARR + (size_t)p.rank * 256), 1u, __ATOMIC_RELAXED, __HIP_MEMORY_SCOPE_AGENT);
        return call;
    }
    if (!sender) return call;
    if (lane < 16) __hip_atomic_store((float*)(dws + OFF_XS) + par * GM + grg * 16 + lane, rinv[lane], __ATOMIC_RELAXED, __HIP_MEMORY_SCOPE_SYSTEM);
    for (int b4 = 8; b4 < NKB; b4 += 4) {
#pragma unroll
        for (int i = 0; i < 4; ++i) {
            const int kb = b4 + i;
            u32x4 dat = *(const LAS u32x4*)(lds + kb * 1024 + lane * 16);
            WAIT_VN(WIN - 1);
            if (kb - WIN >= 8 && slab_start(kb - WIN + 1) && lane == 0)
                __hip_atomic_fetch_add(arr + slab_idx(kb - WIN), 1u, __ATOMIC_RELAXED, __HIP_MEMORY_SCOPE_SYSTEM);
            __builtin_amdgcn_raw_buffer_store_b128(dat, rs, (kb * NRG + grg) * 1024 + lane * 16, 0, 17  );
        }
    }
    WAIT_V(0);
    if (lane == 0) {
#pragma unroll
        for (int kb = NKB - WIN; kb < NKB; ++kb)
            if (kb + 1 == NKB || slab_start(kb + 1))
                __hip_atomic_fetch_add(arr + slab_idx(kb), 1u, __ATOMIC_RELAXED, __HIP_MEMORY_SCOPE_SYSTEM);
    }
    return call;
}

__device__ __forceinline__ void poll(const unsigned* ctr, unsigned target, unsigned* tmo, unsigned call, int lane);
__device__ __forceinline__ void role_wconv(const Params& p, LAS char* lds, int g, unsigned call) {
    const int tid = threadIdx.x, wid = __builtin_amdgcn_readfirstlane(tid >> 6), lane = tid & 63;
    char* myws = p.ws[p.rank];
    const int sl = g >> 5, cq = lane & 15, col0 = (g & 31) * 64 + cq * 4;
    const int kbl = wid * 4 + (lane >> 4), kb = sl * 32 + kbl;
    const float* src = p.w + (size_t)(kb * 32) * NL + col0;
    f32x4 v[32];
#pragma unroll
    for (int r = 0; r < 32; ++r) v[r] = *(const f32x4*)(src + (size_t)r * NL);
    f32x4 am = {0.f, 0.f, 0.f, 0.f};
#pragma unroll
    for (int r = 0; r < 32; ++r)
#pragma unroll
        for (int j = 0; j < 4; ++j) am[j] = fmaxf(am[j], fabsf(v[r][j]));
#pragma unroll
    for (int j = 0; j < 4; ++j) { am[j] = fmaxf(am[j], __shfl_xor(am[j], 16)); am[j] = fmaxf(am[j], __shfl_xor(am[j], 32)); }
    LAS f32x4* red = (LAS f32x4*)(lds + CTL + 1024);
    if (lane < 16) red[wid * 16 + lane] = am;
    __syncthreads();
#pragma unroll
    for (int w = 0; w < 8; ++w) { f32x4 o = red[w * 16 + cq];
#pragma unroll
        for (int j = 0; j < 4; ++j) am[j] = fmaxf(am[j], o[j]); }
    float* pmax = (float*)(myws + OFF_PMAX);
    if (wid == 0) {
        if (lane < 16) {
#pragma unroll
            for (int j = 0; j < 4; ++j) __hip_atomic_store(pmax + sl * NL + col0 + j, am[j], __ATOMIC_RELAXED, __HIP_MEMORY_SCOPE_AGENT);
        }
        WAIT_V(0);
        unsigned* c2 = (unsigned*)(myws + OFF_WC2 + (g & 31) * 128);
        if (lane == 0) __hip_atomic_fetch_add(c2, 1u, __ATOMIC_RELAXED, __HIP_MEMORY_SCOPE_AGENT);
        poll(c2, call * 4u, (unsigned*)(myws + OFF_TMO), call, lane);
    }
    __syncthreads();
#pragma unroll
    for (int q4 = 0; q4 < 4; ++q4)
#pragma unroll
        for (int j = 0; j < 4; ++j) am[j] = fmaxf(am[j], __hip_atomic_load(pmax + q4 * NL + col0 + j, __ATOMIC_RELAXED, __HIP_MEMORY_SCOPE_AGENT));
    float mult[4], inv[4];
#pragma unroll
    for (int j = 0; j < 4; ++j) scale_from_amax(am[j], mult[j], inv[j]);
    if (sl == 0 && tid < 16) {
#pragma unroll
        for (int j = 0; j < 4; ++j) __hip_atomic_store((float*)(myws + OFF_WSC) + col0 + j, inv[j], __ATOMIC_RELAXED, __HIP_MEMORY_SCOPE_AGENT);
    }
    LAS char* fb = lds + wid * 16384;
    const int cgl = cq >> 2, c0 = (cq & 3) * 4;
#pragma unroll
    for (int kq = 0; kq < 4; ++kq)
#pragma unroll
        for (int j = 0; j < 4; ++j) {
            h16x8 h;
#pragma unroll
            for (int r = 0; r < 8; ++r) h[r] = (h16)(v[kq * 8 + r][j] * mult[j]);
            *(LAS h16x8*)(fb + ((lane >> 4) * 4 + cgl) * 1024 + (kq * 16 + c0 + j) * 16) = h;
        }
    asm volatile("s_waitcnt lgkmcnt(0)" ::: "memory");
    __amdgpu_buffer_rsrc_t rs = __builtin_amdgcn_make_buffer_rsrc((void*)(myws + OFF_WT), 0, NL * GK * 2, 0x00020000);
#pragma unroll
    for (int f = 0; f < 16; ++f) {
        const u32x4 dat = *(const LAS u32x4*)(fb + f * 1024 + lane * 16);
        const int kbf = sl * 32 + wid * 4 + (f >> 2), cg = (g & 31) * 4 + (f & 3);
        __builtin_amdgcn_raw_buffer_store_b128(dat, rs, (kbf * NCG + cg) * 1024 + lane * 16, 0, 16  );
    }
    WAIT_V(0);
    __syncthreads();
    if (tid == 0) __hip_atomic_fetch_add((unsigned*)(myws + OFF_WC2 + (g & 31) * 128 + 64), 1u, __ATOMIC_RELAXED, __HIP_MEMORY_SCOPE_AGENT);
}

__device__ __forceinline__ void poll(const unsigned* ctr, unsigned target, unsigned* tmo, unsigned call, int lane) {
    const unsigned long long t0 = __builtin_amdgcn_s_memrealtime();
    while ((int)((unsigned)__builtin_amdgcn_readfirstlane(__hip_atomic_load(ctr, __ATOMIC_RELAXED, __HIP_MEMORY_SCOPE_SYSTEM)) - target) < 0) {
        if ((unsigned)__builtin_amdgcn_readfirstlane(__hip_atomic_load(tmo, __ATOMIC_RELAXED, __HIP_MEMORY_SCOPE_AGENT)) == call
            || __builtin_amdgcn_s_memrealtime() - t0 > DEADLINE_TICKS) {
            if (lane == 0) __hip_atomic_store(tmo, call, __ATOMIC_RELAXED, __HIP_MEMORY_SCOPE_AGENT);
            break;
        }
        __builtin_amdgcn_s_sleep(2);
    }
}

__device__ __forceinline__ void role_gemm(const Params& p, LAS char* lds, int bid, unsigned call) {
    const int tid = threadIdx.x, wid = __builtin_amdgcn_readfirstlane(tid >> 6), lane = tid & 63;
    const int wr = wid >> 1, wc = wid & 1;
    char* myws = p.ws[p.rank];
    const int par = call & 1;
    unsigned* tmo = (unsigned*)(myws + OFF_TMO);

    const int xcd = bid & 7, sl = bid >> 3;
    const int band = 7 * (xcd >> 2) + (sl % 7), pn = 4 * (xcd & 3) + (sl / 7);
    const int rg0 = band * 18 + (band < 4 ? band : 4);
    const int nrg = band < 4 ? 19 : 18;
    const int srcA = rg0 / RG_PER, srcB = (rg0 + nrg - 1) / RG_PER;
    const bool needA = srcA != p.rank, needB = (srcB != srcA) && (srcB != p.rank);
    const bool needL = (srcA == p.rank) || (srcB == p.rank);
    const unsigned* arrA = (const unsigned*)(myws + OFF_ARR + (size_t)srcA * 256);
    const unsigned* arrB = (const unsigned*)(myws + OFF_ARR + (size_t)srcB * 256);
    const unsigned target = call * UNITS;

    const char* gsrc[4];
    int fidx[4];
#pragma unroll
    for (int i = 0; i < 4; ++i) {
        const int f = (i < 3) ? wid * 3 + i : 24 + wid;
        fidx[i] = f;
        const int ra = rg0 + (f < nrg ? f : nrg - 1);
        gsrc[i] = (f < 20) ? myws + OFF_XH + (size_t)par * XH_BYTES + (size_t)ra * 1024 + lane * 16
                           : myws + OFF_WT + (size_t)(pn * 8 + ((f - 20) & 7)) * 1024 + lane * 16;
    }
#define STAGE(kb, slot) do { _Pragma("unroll") for (int _i = 0; _i < 4; ++_i) { if (_i < 3 || wid < 4) \
        __builtin_amdgcn_global_load_lds((const unsigned*)(gsrc[_i] + (size_t)(kb) * ((fidx[_i] < 20) ? NRG * 1024 : NCG * 1024)), \
            (LAS unsigned*)(lds + (slot) * STG + fidx[_i] * 1024), 16, 0, 0); } } while (0)
#define BAR __builtin_amdgcn_s_barrier()
#define WAIT_STAGES(n) do { if (wid < 4) WAIT_VN((n) * 4); else WAIT_VN((n) * 3); } while (0)
    f32x4 acc[5][4];
#pragma unroll
    for (int m = 0; m < 5; ++m)
#pragma unroll
        for (int n = 0; n < 4; ++n) acc[m][n] = (f32x4){0.f, 0.f, 0.f, 0.f};
    h16x8 A0[5], B0[4], A1[5], B1[4];
    const int fbase = (wr < 3) ? wr * 5 : nrg - 4;
    const int fcnt = (wr < 2) ? 5 : ((wr == 2) ? nrg - 14 : 4);
    const bool full5 = fcnt == 5;
    const int aoff = fbase * 1024 + lane * 16, boff = (20 + wc * 4) * 1024 + lane * 16;
#define LOADF(Af, Bf, slot) do { const LAS char* _b = lds + (slot) * STG; \
        _Pragma("unroll") for (int n = 0; n < 4; ++n) Bf[n] = *(const LAS h16x8*)(_b + boff + n * 1024); \
        _Pragma("unroll") for (int m = 0; m < 5; ++m) Af[m] = *(const LAS h16x8*)(_b + aoff + m * 1024); } while (0)
#define MMA(Af, Bf, m0, m1) do { _Pragma("unroll") for (int m = m0; m < m1; ++m) _Pragma("unroll") for (int n = 0; n < 4; ++n) \
        acc[m][n] = __builtin_amdgcn_mfma_f32_16x16x32_f16(Bf[n], Af[m], acc[m][n], 0, 0, 0); } while (0)
#define SCHEDULE do { _Pragma("unroll") for (int _q = 0; _q < 9; ++_q) { __builtin_amdgcn_sched_group_barrier(0x8, 1, 0); __builtin_amdgcn_sched_group_barrier(0x100, 1, 0); } \
        __builtin_amdgcn_sched_group_barrier(0x8, 3, 0); } while (0)
    int rdy = (needA || needB) ? 0 : NKB;
    const int gl = lane & 31;
    const bool gact = gl < NSLAB && ((lane < 32) ? needA : needB);
    const unsigned* gptr = ((lane < 32) ? arrA : arrB) + gl;
#define GATE(kb) do { if ((kb) >= rdy) { const unsigned long long t0 = __builtin_amdgcn_s_memrealtime(); \
        for (;;) { const unsigned v = __hip_atomic_load(gptr, __ATOMIC_RELAXED, __HIP_MEMORY_SCOPE_SYSTEM); \
            const unsigned long long bad_ = __ballot(gact && (int)(v - target) < 0); \
            const unsigned both = (unsigned)bad_ | (unsigned)(bad_ >> 32); \
            rdy = both ? slab_first(__builtin_ctz(both)) : NKB; \
            if ((kb) < rdy) break; \
            if ((unsigned)__builtin_amdgcn_readfirstlane(__hip_atomic_load(tmo, __ATOMIC_RELAXED, __HIP_MEMORY_SCOPE_AGENT)) == call \
                || __builtin_amdgcn_s_memrealtime() - t0 > DEADLINE_TICKS) { \
                if (lane == 0) __hip_atomic_store(tmo, call, __ATOMIC_RELAXED, __HIP_MEMORY_SCOPE_AGENT); rdy = NKB; break; } \
            __builtin_amdgcn_s_sleep(1); } } } while (0)

    if (wid == 0) {
        poll((const unsigned*)(myws + OFF_WC2 + (2 * pn) * 128 + 64), call * 4u, tmo, call, lane);
        poll((const unsigned*)(myws + OFF_WC2 + (2 * pn + 1) * 128 + 64), call * 4u, tmo, call, lane);
        if (needL) poll((const unsigned*)(myws + OFF_ARR + (size_t)p.rank * 256), target, tmo, call, lane);
        GATE(0);
        __builtin_amdgcn_fence(__ATOMIC_ACQUIRE, "");
        WAIT_V(0);
    }
    asm volatile("" ::: "memory"); BAR; asm volatile("" ::: "memory");
    const int fr = lane & 15, fq = lane >> 4;
    f32x4 cs[4];
    float rsc[5];
    {
        const float* xs = (const float*)(myws + OFF_XS0) + par * GM;
        const float* wsc = (const float*)(myws + OFF_WSC) + pn * 128 + wc * 64 + 4 * fq;
#pragma unroll
        for (int n = 0; n < 4; ++n)
#pragma unroll
            for (int jj = 0; jj < 4; ++jj) cs[n][jj] = __hip_atomic_load(wsc + n * 16 + jj, __ATOMIC_RELAXED, __HIP_MEMORY_SCOPE_AGENT);
#pragma unroll
        for (int m = 0; m < 5; ++m) {
            const int f = fbase + (m < fcnt ? m : 0);
            rsc[m] = __hip_atomic_load(xs + (rg0 + f) * 16 + fr, __ATOMIC_RELAXED, __HIP_MEMORY_SCOPE_SYSTEM);
        }
        WAIT_V(0);
    }
#pragma unroll
    for (int i = 0; i < NS; ++i) STAGE(i, i);
    WAIT_STAGES(NS - 1); BAR;
    LOADF(A0, B0, 0);
    WAIT_STAGES(NS - 2);
    asm volatile("s_waitcnt lgkmcnt(0)" ::: "memory");
    BAR;
    int sj = 0;
    for (int j = 0; j < NKB; j += 2) {
        unsigned pv = 0;
        const bool polling = (wid == 0) && (rdy < NKB) && (rdy <= j + 16);
        if (polling) asm volatile("global_load_dword %0, %1, off sc0 sc1" : "=v"(pv) : "v"(gptr) : "memory");
        const int s1 = (sj + 1 == NS) ? 0 : sj + 1, s2 = (s1 + 1 == NS) ? 0 : s1 + 1;
        LOADF(A1, B1, s1);
        MMA(A0, B0, 0, 3);
        SCHEDULE;
        { const int kn = (j + NS < NKB) ? j + NS : 0; STAGE(kn, sj); }
        MMA(A0, B0, 3, 4);
        if (full5) MMA(A0, B0, 4, 5);
        WAIT_STAGES(NS - 2);
        asm volatile("s_waitcnt lgkmcnt(0)" ::: "memory");
        BAR;
        LOADF(A0, B0, s2);
        MMA(A1, B1, 0, 3);
        SCHEDULE;
        { const int kn = (j + 1 + NS < NKB) ? j + 1 + NS : 0; STAGE(kn, s1); }
        MMA(A1, B1, 3, 4);
        if (full5) MMA(A1, B1, 4, 5);
        if (j == 6) {
            const float* xs = (const float*)(myws + OFF_XS) + par * GM;
#pragma unroll
            for (int m = 0; m < 5; ++m) {
                const int f = fbase + (m < fcnt ? m : 0);
                const float full = __hip_atomic_load(xs + (rg0 + f) * 16 + fr, __ATOMIC_RELAXED, __HIP_MEMORY_SCOPE_SYSTEM);
                const float ratio = rsc[m] / full;
                rsc[m] = full;
#pragma unroll
                for (int n = 0; n < 4; ++n) acc[m][n] *= ratio;
            }
        }
        WAIT_STAGES(NS - 2);
        sj = s2;
        if (polling) {
            WAIT_VN(8);
            asm volatile("" : "+v"(pv) :: "memory");
            const unsigned long long bad_ = __ballot(gact && (int)(pv - target) < 0);
            const unsigned both = (unsigned)bad_ | (unsigned)(bad_ >> 32);
            const int r2 = both ? slab_first(__builtin_ctz(both)) : NKB;
            rdy = r2 > rdy ? r2 : rdy;
            constexpr int GA = (NS & 1) ? NS + 3 : NS + 2;
            if (j + GA < NKB && slab_start(j + GA)) {
                GATE(j + GA);
            }
        }
        asm volatile("s_waitcnt lgkmcnt(0)" ::: "memory");
        BAR;
    }
    WAIT_V(0);
    BAR;
    const bool bad = __hip_atomic_load(tmo, __ATOMIC_RELAXED, __HIP_MEMORY_SCOPE_AGENT) == call;
    const float qnan = __builtin_nanf("");
    LAS char* eb = lds + wid * 8704;
#pragma unroll
    for (int m = 0; m < 5; ++m) {
        const int f = fbase + m;
        if (m < fcnt) {
            LAS char* b = eb + (m & 1) * 4352;
#pragma unroll
            for (int n = 0; n < 4; ++n) {
                f32x4 o = (acc[m][n] * rsc[m]) * cs[n];
#pragma unroll
                for (int jj = 0; jj < 4; ++jj) o[jj] = bad ? qnan : fmaxf(o[jj], 0.f);
                *(LAS f32x4*)(b + fr * 272 + n * 64 + fq * 16) = o;
            }
            asm volatile("s_waitcnt lgkmcnt(0)" ::: "memory");
            float* op = p.out + (size_t)((rg0 + f) * 16 + fq) * NL + pn * 128 + wc * 64 + fr * 4;
#pragma unroll
            for (int i = 0; i < 4; ++i) {
                const f32x4 o = *(const LAS f32x4*)(b + (i * 4 + fq) * 272 + fr * 16);
                __builtin_nontemporal_store(o, (f32x4*)(op + (size_t)(i * 4) * NL));
            }
        }
    }
#undef STAGE
#undef BAR
#undef LOADF
#undef MMA
#undef GATE
#undef SCHEDULE
#undef WAIT_STAGES
}

__global__ void __launch_bounds__(512, 2) ag_gemm_kernel(Params p) {
    extern __shared__ __attribute__((aligned(16))) char smem[];
    LAS char* lds = (LAS char*)smem;
    const int b = blockIdx.x;
    int role;
    if (b < PUSH_BLOCKS) { role = 3; role_push(p, lds, b, false); }
    else {
        role = 2;
        const int g = b - PUSH_BLOCKS;
        unsigned call;
        if (g >= WCONV_BLOCKS && g < WCONV_BLOCKS + RG_PER) call = role_push(p, lds, g - WCONV_BLOCKS, true);
        else {
            call = draw_call((unsigned*)(p.ws[p.rank] + OFF_CTR), (LAS unsigned*)(lds + CTL));
            if (g < WCONV_BLOCKS) role_wconv(p, lds, g, call);
        }
        role_gemm(p, lds, g, call);
    }
}

}

extern "C" void kernel_launch(int rank, int nranks, ncclComm_t comm,
                              hipStream_t stream,
                              void* const* d_in, const int* in_sizes, int n_in,
                              void* d_out, int out_size,
                              const int* in_dtypes, int out_dtype,
                              void* const* d_out_all,
                              void* const* d_ws_all, size_t ws_bytes) {
    static bool init = false;
    if (nranks != NR || ws_bytes < WS_NEED) { fprintf(stderr, "unexpected configuration\n"); return; }
    if (!init) {
        HIP_CHECK(hipFuncSetAttribute((const void*)ag_gemm_kernel, hipFuncAttributeMaxDynamicSharedMemorySize, LDS_BYTES));
        init = true;
    }
    Params p;
    p.x = (const float*)d_in[0];
    p.w = (const float*)d_in[1];
    p.out = (float*)d_out;
    for (int i = 0; i < NR; ++i) p.ws[i] = (char*)d_ws_all[i];
    p.rank = rank;
    p.phase = 0;
    ag_gemm_kernel<<<GEMM_BLOCKS + PUSH_BLOCKS, 512, LDS_BYTES, stream>>>(p);
    HIP_CHECK(hipGetLastError());
}
```

```cpp
#include <hip/hip_runtime.h>
#include <rccl/rccl.h>
#include <cstdio>
#include <cstdint>

#define HIP_CHECK(expr) do { hipError_t _e = (expr); if (_e != hipSuccess) { \
    fprintf(stderr, "HIP error %s at %s:%d\n", hipGetErrorString(_e), __FILE__, __LINE__); \
    return; } } while (0)

namespace {

typedef _Float16 h16;
typedef _Float16 h16x8 __attribute__((ext_vector_type(8)));
typedef float f32x4 __attribute__((ext_vector_type(4)));
typedef unsigned u32x4 __attribute__((ext_vector_type(4)));
#define LAS __attribute__((address_space(3)))

constexpr int EC = 1;
constexpr int WIN = 5;
constexpr int WIN2 = 3;
constexpr int KSW = 32;
constexpr unsigned long long PACE_T = 60, PACE_0 = 400;

constexpr int NR = 8;
constexpr int GM = 4096;
constexpr int GK = 4096;
constexpr int NL = 2048;
constexpr int MS = GM / NR;
constexpr int NKB = GK / 32;
constexpr int NRG = GM / 16;
constexpr int NCG = NL / 16;
constexpr int RG_PER = MS / 16;

constexpr int UNITS = RG_PER;
__host__ __device__ constexpr bool slab_start(int kb) { return kb < 64 ? (kb & 7) == 0 : (kb < 112 ? (kb & 3) == 0 : (kb & 1) == 0); }
constexpr int NSLAB = 28;
__host__ __device__ constexpr int slab_first(int s) { return s < 8 ? s * 8 : (s < 20 ? 64 + (s - 8) * 4 : 112 + (s - 20) * 2); }
__host__ __device__ constexpr int slab_idx(int kb) { return kb < 64 ? kb >> 3 : (kb < 112 ? 8 + ((kb - 64) >> 2) : 20 + ((kb - 112) >> 1)); }

constexpr int PUSH_BLOCKS = RG_PER;
constexpr int WCONV_BLOCKS = NKB;
constexpr int GEMM_BLOCKS = 224;

constexpr size_t OFF_CTR = 0;
constexpr size_t OFF_TMO = 1024;
constexpr size_t OFF_WC2 = 96 << 10;
constexpr size_t OFF_PMAX = 256 << 10;
constexpr size_t OFF_ARR = 4096;
constexpr size_t OFF_XS  = 128 << 10;
constexpr size_t OFF_XS0 = 160 << 10;
constexpr size_t OFF_WSC = 192 << 10;
constexpr size_t OFF_XH  = 1 << 20;
constexpr size_t XH_BYTES = (size_t)GM * GK * 2;
constexpr size_t OFF_WT  = OFF_XH + 2 * XH_BYTES;
constexpr size_t WS_NEED = OFF_WT + (size_t)NL * GK * 2;

constexpr unsigned long long DEADLINE_TICKS = 50000000ull;

constexpr int NS = 4;
constexpr int STG = 28 * 1024;
constexpr int CTL = (NS * STG > 131072) ? NS * STG : 131072;
constexpr int LDS_BYTES = CTL + 4096;

struct Params {
    const float* x;
    const float* w;
    float* out;
    char* ws[NR];
    int rank;
};

__device__ __forceinline__ unsigned draw_call(unsigned* ctr, LAS unsigned* slot) {
    if (threadIdx.x == 0) {
        unsigned t = __hip_atomic_fetch_add(ctr, 1u, __ATOMIC_RELAXED, __HIP_MEMORY_SCOPE_AGENT);
        *slot = t / gridDim.x + 1u;
    }
    __syncthreads();
    unsigned c = __builtin_amdgcn_readfirstlane(*slot);
    __syncthreads();
    return c;
}

__device__ __forceinline__ void scale_from_amax(float amax, float& mult, float& inv) {
    int ef = (int)((__float_as_uint(amax) >> 23) & 0xffu);
    int e = ef - 127 - 14;
    if (ef == 0xff || amax == 0.f) e = 0;
    e = e < -100 ? -100 : (e > 113 ? 113 : e);
    mult = __uint_as_float((unsigned)(127 - e) << 23);
    inv  = __uint_as_float((unsigned)(127 + e) << 23);
}

__device__ __forceinline__ float amax4(f32x4 v) {
    return fmaxf(fmaxf(fabsf(v[0]), fabsf(v[1])), fmaxf(fabsf(v[2]), fabsf(v[3])));
}

#define WAIT_V(n) asm volatile("s_waitcnt vmcnt(" #n ")" ::: "memory")
#define WAIT_VN(n) asm volatile("s_waitcnt vmcnt(%0)" :: "n"(n) : "memory")

__device__ __forceinline__ unsigned role_push(const Params& p, LAS char* lds, int rg, bool local) {
    typedef _Float16 h16x4 __attribute__((ext_vector_type(4)));
    const int tid = threadIdx.x, wid = __builtin_amdgcn_readfirstlane(tid >> 6), lane = tid & 63;
    const unsigned long long tstart = __builtin_amdgcn_s_memrealtime();
    unsigned ticket = 0;
    if (tid == 0) ticket = __hip_atomic_fetch_add((unsigned*)(p.ws[p.rank] + OFF_CTR), 1u, __ATOMIC_RELAXED, __HIP_MEMORY_SCOPE_AGENT);
    const float* xb = p.x + (size_t)rg * 16 * GK;
    LAS float* rinv = (LAS float*)(lds + CTL + 64);
    f32x4 v[2][16];
#pragma unroll
    for (int c = 0; c < 16; ++c)
#pragma unroll
        for (int r = 0; r < 2; ++r) v[r][c] = *(const f32x4*)(xb + (size_t)(wid * 2 + r) * GK + lane * 4 + c * 256);
    const int kl = lane * 4;
    const int loff = (kl >> 5) * 1024 + (((kl >> 3) & 3) * 16) * 16 + (kl & 4) * 2;
#pragma unroll
    for (int r = 0; r < 2; ++r) {
        const int row = wid * 2 + r;
        float am = 0.f;
#pragma unroll
        for (int c = 0; c < EC; ++c) am = fmaxf(am, amax4(v[r][c]));
#pragma unroll
        for (int o = 32; o >= 1; o >>= 1) am = fmaxf(am, __shfl_xor(am, o));
        float mult, inv;
        scale_from_amax(am, mult, inv);
        if (lane == 0) rinv[16 + row] = inv;
#pragma unroll
        for (int c = 0; c < EC; ++c) {
            h16x4 h;
#pragma unroll
            for (int j = 0; j < 4; ++j) h[j] = (h16)(v[r][c][j] * mult);
            *(LAS h16x4*)(lds + c * 8192 + loff + row * 16) = h;
        }
    }
    if (tid == 0) *(LAS unsigned*)(lds + CTL) = ticket / gridDim.x + 1u;
    __syncthreads();
    const unsigned call = __builtin_amdgcn_readfirstlane(*(LAS unsigned*)(lds + CTL));
    const int par = call & 1;
    const int grg = p.rank * RG_PER + rg;
    const bool sender = wid < 7;
    char* dws = p.ws[(p.rank + 1 + wid) & 7];
    __amdgpu_buffer_rsrc_t rs = __builtin_amdgcn_make_buffer_rsrc(
        (void*)(dws + OFF_XH + (size_t)par * XH_BYTES), 0, (int)XH_BYTES, 0x00020000);
    unsigned* arr = (unsigned*)(dws + OFF_ARR + (size_t)p.rank * 256);
    if (local && sender) {
        if (lane < 16) __hip_atomic_store((float*)(dws + OFF_XS0) + par * GM + grg * 16 + lane, rinv[16 + lane], __ATOMIC_RELAXED, __HIP_MEMORY_SCOPE_SYSTEM);
#pragma unroll
        for (int kb = 0; kb < 8 * EC; ++kb) {
            u32x4 dat = *(const LAS u32x4*)(lds + kb * 1024 + lane * 16);
            __builtin_amdgcn_raw_buffer_store_b128(dat, rs, (kb * NRG + grg) * 1024 + lane * 16, 0, 17  );
        }
    }
#pragma unroll
    for (int r = 0; r < 2; ++r) {
        const int row = wid * 2 + r;
        float am = 0.f;
#pragma unroll
        for (int c = 0; c < 16; ++c) am = fmaxf(am, amax4(v[r][c]));
#pragma unroll
        for (int o = 32; o >= 1; o >>= 1) am = fmaxf(am, __shfl_xor(am, o));
        float mult, inv;
        scale_from_amax(am, mult, inv);
        if (lane == 0) rinv[row] = inv;
#pragma unroll
        for (int c = EC; c < 16; ++c) {
            h16x4 h;
#pragma unroll
            for (int j = 0; j < 4; ++j) h[j] = (h16)(v[r][c][j] * mult);
            *(LAS h16x4*)(lds + c * 8192 + loff + row * 16) = h;
        }
    }
    if (local && sender) {
        WAIT_V(0);
        if (lane < EC) __hip_atomic_fetch_add(arr + lane, 1u, __ATOMIC_RELAXED, __HIP_MEMORY_SCOPE_SYSTEM);
    }
    __syncthreads();
    if (local) {
        dws = p.ws[p.rank];
        rs = __builtin_amdgcn_make_buffer_rsrc((void*)(dws + OFF_XH + (size_t)par * XH_BYTES), 0, (int)XH_BYTES, 0x00020000);
        if (tid < 16) __hip_atomic_store((float*)(dws + OFF_XS) + par * GM + grg * 16 + tid, rinv[tid], __ATOMIC_RELAXED, __HIP_MEMORY_SCOPE_AGENT);
        else if (tid < 32) __hip_atomic_store((float*)(dws + OFF_XS0) + par * GM + grg * 16 + (tid - 16), rinv[tid], __ATOMIC_RELAXED, __HIP_MEMORY_SCOPE_AGENT);
#pragma unroll
        for (int i = 0; i < 16; ++i) {
            const int kb = i * 8 + wid;
            u32x4 dat = *(const LAS u32x4*)(lds + kb * 1024 + lane * 16);
            __builtin_amdgcn_raw_buffer_store_b128(dat, rs, (kb * NRG + grg) * 1024 + lane * 16, 0, 16  );
        }
        WAIT_V(0);
        __syncthreads();
        if (tid == 0) __hip_atomic_fetch_add((unsigned*)(dws + OFF_ARR + (size_t)p.rank * 256), 1u, __ATOMIC_RELAXED, __HIP_MEMORY_SCOPE_AGENT);
        return call;
    }
    if (!sender) return call;
    if (lane < 16) __hip_atomic_store((float*)(dws + OFF_XS) + par * GM + grg * 16 + lane, rinv[lane], __ATOMIC_RELAXED, __HIP_MEMORY_SCOPE_SYSTEM);
    int sig = 8 * EC;
#define PACE(kb) do { const unsigned long long due = tstart + PACE_0 + (unsigned long long)(kb) * PACE_T; \
        if (__builtin_amdgcn_s_memrealtime() < due) { \
            WAIT_V(0); \
            for (; sig < (kb); ++sig) \
                if (slab_start(sig + 1) && lane == 0) __hip_atomic_fetch_add(arr + slab_idx(sig), 1u, __ATOMIC_RELAXED, __HIP_MEMORY_SCOPE_SYSTEM); \
            while (__builtin_amdgcn_s_memrealtime() < due) __builtin_amdgcn_s_sleep(1); } } while (0)
#define PUSH_RANGE(k0, k1, W) \
    for (int b4 = (k0); b4 < (k1); b4 += 4) { \
        PACE(b4); \
        _Pragma("unroll") for (int i = 0; i < 4; ++i) { \
            const int kb = b4 + i; \
            u32x4 dat = *(const LAS u32x4*)(lds + kb * 1024 + lane * 16); \
            WAIT_VN((W) - 1);                               \
            for (; sig <= kb - (W); ++sig) \
                if (slab_start(sig + 1) && lane == 0) __hip_atomic_fetch_add(arr + slab_idx(sig), 1u, __ATOMIC_RELAXED, __HIP_MEMORY_SCOPE_SYSTEM); \
            __builtin_amdgcn_raw_buffer_store_b128(dat, rs, (kb * NRG + grg) * 1024 + lane * 16, 0, 17  ); \
        } \
    }
    PUSH_RANGE(8 * EC, KSW, WIN)
    PUSH_RANGE(KSW, NKB, WIN2)
#undef PUSH_RANGE
#undef PACE
    WAIT_V(0);
    for (; sig < NKB; ++sig)
        if ((sig + 1 == NKB || slab_start(sig + 1)) && lane == 0) __hip_atomic_fetch_add(arr + slab_idx(sig), 1u, __ATOMIC_RELAXED, __HIP_MEMORY_SCOPE_SYSTEM);
    return call;
}

__device__ __forceinline__ void poll(const unsigned* ctr, unsigned target, unsigned* tmo, unsigned call, int lane);
__device__ __forceinline__ void role_wconv(const Params& p, LAS char* lds, int g, unsigned call) {
    const int tid = threadIdx.x, wid = __builtin_amdgcn_readfirstlane(tid >> 6), lane = tid & 63;
    char* myws = p.ws[p.rank];
    const int sl = g >> 5, cq = lane & 15, col0 = (g & 31) * 64 + cq * 4;
    const int kbl = wid * 4 + (lane >> 4), kb = sl * 32 + kbl;
    const float* src = p.w + (size_t)(kb * 32) * NL + col0;
    f32x4 v[32];
#pragma unroll
    for (int r = 0; r < 32; ++r) v[r] = *(const f32x4*)(src + (size_t)r * NL);
    f32x4 am = {0.f, 0.f, 0.f, 0.f};
#pragma unroll
    for (int r = 0; r < 32; ++r)
#pragma unroll
        for (int j = 0; j < 4; ++j) am[j] = fmaxf(am[j], fabsf(v[r][j]));
#pragma unroll
    for (int j = 0; j < 4; ++j) { am[j] = fmaxf(am[j], __shfl_xor(am[j], 16)); am[j] = fmaxf(am[j], __shfl_xor(am[j], 32)); }
    LAS f32x4* red = (LAS f32x4*)(lds + CTL + 1024);
    if (lane < 16) red[wid * 16 + lane] = am;
    __syncthreads();
#pragma unroll
    for (int w = 0; w < 8; ++w) { f32x4 o = red[w * 16 + cq];
#pragma unroll
        for (int j = 0; j < 4; ++j) am[j] = fmaxf(am[j], o[j]); }
    float* pmax = (float*)(myws + OFF_PMAX);
    if (wid == 0) {
        if (lane < 16) {
#pragma unroll
            for (int j = 0; j < 4; ++j) __hip_atomic_store(pmax + sl * NL + col0 + j, am[j], __ATOMIC_RELAXED, __HIP_MEMORY_SCOPE_AGENT);
        }
        WAIT_V(0);
        unsigned* c2 = (unsigned*)(myws + OFF_WC2 + (g & 31) * 128);
        if (lane == 0) __hip_atomic_fetch_add(c2, 1u, __ATOMIC_RELAXED, __HIP_MEMORY_SCOPE_AGENT);
        poll(c2, call * 4u, (unsigned*)(myws + OFF_TMO), call, lane);
    }
    __syncthreads();
#pragma unroll
    for (int q4 = 0; q4 < 4; ++q4)
#pragma unroll
        for (int j = 0; j < 4; ++j) am[j] = fmaxf(am[j], __hip_atomic_load(pmax + q4 * NL + col0 + j, __ATOMIC_RELAXED, __HIP_MEMORY_SCOPE_AGENT));
    float mult[4], inv[4];
#pragma unroll
    for (int j = 0; j < 4; ++j) scale_from_amax(am[j], mult[j], inv[j]);
    if (sl == 0 && tid < 16) {
#pragma unroll
        for (int j = 0; j < 4; ++j) __hip_atomic_store((float*)(myws + OFF_WSC) + col0 + j, inv[j], __ATOMIC_RELAXED, __HIP_MEMORY_SCOPE_AGENT);
    }
    LAS char* fb = lds + wid * 16384;
    const int cgl = cq >> 2, c0 = (cq & 3) * 4;
#pragma unroll
    for (int kq = 0; kq < 4; ++kq)
#pragma unroll
        for (int j = 0; j < 4; ++j) {
            h16x8 h;
#pragma unroll
            for (int r = 0; r < 8; ++r) h[r] = (h16)(v[kq * 8 + r][j] * mult[j]);
            *(LAS h16x8*)(fb + ((lane >> 4) * 4 + cgl) * 1024 + (kq * 16 + c0 + j) * 16) = h;
        }
    asm volatile("s_waitcnt lgkmcnt(0)" ::: "memory");
    __amdgpu_buffer_rsrc_t rs = __builtin_amdgcn_make_buffer_rsrc((void*)(myws + OFF_WT), 0, NL * GK * 2, 0x00020000);
#pragma unroll
    for (int f = 0; f < 16; ++f) {
        const u32x4 dat = *(const LAS u32x4*)(fb + f * 1024 + lane * 16);
        const int kbf = sl * 32 + wid * 4 + (f >> 2), cg = (g & 31) * 4 + (f & 3);
        __builtin_amdgcn_raw_buffer_store_b128(dat, rs, (kbf * NCG + cg) * 1024 + lane * 16, 0, 16  );
    }
    WAIT_V(0);
    __syncthreads();
    if (tid == 0) __hip_atomic_fetch_add((unsigned*)(myws + OFF_WC2 + (g & 31) * 128 + 64), 1u, __ATOMIC_RELAXED, __HIP_MEMORY_SCOPE_AGENT);
}

__device__ __forceinline__ void poll(const unsigned* ctr, unsigned target, unsigned* tmo, unsigned call, int lane) {
    const unsigned long long t0 = __builtin_amdgcn_s_memrealtime();
    while ((int)((unsigned)__builtin_amdgcn_readfirstlane(__hip_atomic_load(ctr, __ATOMIC_RELAXED, __HIP_MEMORY_SCOPE_SYSTEM)) - target) < 0) {
        if ((unsigned)__builtin_amdgcn_readfirstlane(__hip_atomic_load(tmo, __ATOMIC_RELAXED, __HIP_MEMORY_SCOPE_AGENT)) == call
            || __builtin_amdgcn_s_memrealtime() - t0 > DEADLINE_TICKS) {
            if (lane == 0) __hip_atomic_store(tmo, call, __ATOMIC_RELAXED, __HIP_MEMORY_SCOPE_AGENT);
            break;
        }
        __builtin_amdgcn_s_sleep(2);
    }
}

__device__ __forceinline__ void role_gemm(const Params& p, LAS char* lds, int bid, unsigned call) {
    const int tid = threadIdx.x, wid = __builtin_amdgcn_readfirstlane(tid >> 6), lane = tid & 63;
    const int wr = wid >> 1, wc = wid & 1;
    char* myws = p.ws[p.rank];
    const int par = call & 1;
    unsigned* tmo = (unsigned*)(myws + OFF_TMO);

    const int xcd = bid & 7, sl = bid >> 3;
    const int band = 7 * (xcd >> 2) + (sl % 7), pn = 4 * (xcd & 3) + (sl / 7);
    const int rg0 = band * 18 + (band < 4 ? band : 4);
    const int nrg = band < 4 ? 19 : 18;
    const int srcA = rg0 / RG_PER, srcB = (rg0 + nrg - 1) / RG_PER;
    const bool needA = srcA != p.rank, needB = (srcB != srcA) && (srcB != p.rank);
    const bool needL = (srcA == p.rank) || (srcB == p.rank);
    const unsigned* arrA = (const unsigned*)(myws + OFF_ARR + (size_t)srcA * 256);
    const unsigned* arrB = (const unsigned*)(myws + OFF_ARR + (size_t)srcB * 256);
    const unsigned target = call * UNITS;

    const char* gsrc[4];
    int fidx[4];
#pragma unroll
    for (int i = 0; i < 4; ++i) {
        const int f = (i < 3) ? wid * 3 + i : 24 + wid;
        fidx[i] = f;
        const int ra = rg0 + (f < nrg ? f : nrg - 1);
        gsrc[i] = (f < 20) ? myws + OFF_XH + (size_t)par * XH_BYTES + (size_t)ra * 1024 + lane * 16
                           : myws + OFF_WT + (size_t)(pn * 8 + ((f - 20) & 7)) * 1024 + lane * 16;
    }
#define STAGE(kb, slot) do { _Pragma("unroll") for (int _i = 0; _i < 4; ++_i) { if (_i < 3 || wid < 4) \
        __builtin_amdgcn_global_load_lds((const unsigned*)(gsrc[_i] + (size_t)(kb) * ((fidx[_i] < 20) ? NRG * 1024 : NCG * 1024)), \
            (LAS unsigned*)(lds + (slot) * STG + fidx[_i] * 1024), 16, 0, 0); } } while (0)
#define BAR __builtin_amdgcn_s_barrier()
#define WAIT_STAGES(n) do { if (wid < 4) WAIT_VN((n) * 4); else WAIT_VN((n) * 3); } while (0)
    f32x4 acc[5][4];
#pragma unroll
    for (int m = 0; m < 5; ++m)
#pragma unroll
        for (int n = 0; n < 4; ++n) acc[m][n] = (f32x4){0.f, 0.f, 0.f, 0.f};
    h16x8 A0[5], B0[4], A1[5], B1[4];
    const int fbase = (wr < 3) ? wr * 5 : nrg - 4;
    const int fcnt = (wr < 2) ? 5 : ((wr == 2) ? nrg - 14 : 4);
    const bool full5 = fcnt == 5;
    const int aoff = fbase * 1024 + lane * 16, boff = (20 + wc * 4) * 1024 + lane * 16;
#define LOADF(Af, Bf, slot) do { const LAS char* _b = lds + (slot) * STG; \
        _Pragma("unroll") for (int n = 0; n < 4; ++n) Bf[n] = *(const LAS h16x8*)(_b + boff + n * 1024); \
        _Pragma("unroll") for (int m = 0; m < 5; ++m) Af[m] = *(const LAS h16x8*)(_b + aoff + m * 1024); } while (0)
#define MMA(Af, Bf, m0, m1) do { _Pragma("unroll") for (int m = m0; m < m1; ++m) _Pragma("unroll") for (int n = 0; n < 4; ++n) \
        acc[m][n] = __builtin_amdgcn_mfma_f32_16x16x32_f16(Bf[n], Af[m], acc[m][n], 0, 0, 0); } while (0)
#define SCHEDULE do { _Pragma("unroll") for (int _q = 0; _q < 9; ++_q) { __builtin_amdgcn_sched_group_barrier(0x8, 1, 0); __builtin_amdgcn_sched_group_barrier(0x100, 1, 0); } \
        __builtin_amdgcn_sched_group_barrier(0x8, 3, 0); } while (0)
    int rdy = (needA || needB) ? 0 : NKB;
    const int gl = lane & 31;
    const bool gact = gl < NSLAB && ((lane < 32) ? needA : needB);
    const unsigned* gptr = ((lane < 32) ? arrA : arrB) + gl;
#define GATE(kb) do { if ((kb) >= rdy) { const unsigned long long t0 = __builtin_amdgcn_s_memrealtime(); \
        for (;;) { const unsigned v = __hip_atomic_load(gptr, __ATOMIC_RELAXED, __HIP_MEMORY_SCOPE_SYSTEM); \
            const unsigned long long bad_ = __ballot(gact && (int)(v - target) < 0); \
            const unsigned both = (unsigned)bad_ | (unsigned)(bad_ >> 32); \
            rdy = both ? slab_first(__builtin_ctz(both)) : NKB; \
            if ((kb) < rdy) break; \
            if ((unsigned)__builtin_amdgcn_readfirstlane(__hip_atomic_load(tmo, __ATOMIC_RELAXED, __HIP_MEMORY_SCOPE_AGENT)) == call \
                || __builtin_amdgcn_s_memrealtime() - t0 > DEADLINE_TICKS) { \
                if (lane == 0) __hip_atomic_store(tmo, call, __ATOMIC_RELAXED, __HIP_MEMORY_SCOPE_AGENT); rdy = NKB; break; } \
            __builtin_amdgcn_s_sleep(1); } } } while (0)

    if (wid == 0) {
        poll((const unsigned*)(myws + OFF_WC2 + (2 * pn) * 128 + 64), call * 4u, tmo, call, lane);
        poll((const unsigned*)(myws + OFF_WC2 + (2 * pn + 1) * 128 + 64), call * 4u, tmo, call, lane);
        if (needL) poll((const unsigned*)(myws + OFF_ARR + (size_t)p.rank * 256), target, tmo, call, lane);
        GATE(0);
        __builtin_amdgcn_fence(__ATOMIC_ACQUIRE, "");
        WAIT_V(0);
    }
    asm volatile("" ::: "memory"); BAR; asm volatile("" ::: "memory");
    const int fr = lane & 15, fq = lane >> 4;
    f32x4 cs[4];
    float rsc[5];
    {
        const float* xs = (const float*)(myws + OFF_XS0) + par * GM;
        const float* wsc = (const float*)(myws + OFF_WSC) + pn * 128 + wc * 64 + 4 * fq;
#pragma unroll
        for (int n = 0; n < 4; ++n)
#pragma unroll
            for (int jj = 0; jj < 4; ++jj) cs[n][jj] = __hip_atomic_load(wsc + n * 16 + jj, __ATOMIC_RELAXED, __HIP_MEMORY_SCOPE_AGENT);
#pragma unroll
        for (int m = 0; m < 5; ++m) {
            const int f = fbase + (m < fcnt ? m : 0);
            rsc[m] = __hip_atomic_load(xs + (rg0 + f) * 16 + fr, __ATOMIC_RELAXED, __HIP_MEMORY_SCOPE_SYSTEM);
        }
        WAIT_V(0);
    }
#pragma unroll
    for (int i = 0; i < NS; ++i) STAGE(i, i);
    WAIT_STAGES(NS - 1); BAR;
    LOADF(A0, B0, 0);
    WAIT_STAGES(NS - 2);
    asm volatile("s_waitcnt lgkmcnt(0)" ::: "memory");
    BAR;
    int sj = 0;
    for (int j = 0; j < NKB; j += 2) {
        unsigned pv = 0;
        const bool polling = (wid == 0) && (rdy < NKB) && (rdy <= j + 16);
        if (polling) asm volatile("global_load_dword %0, %1, off sc0 sc1" : "=v"(pv) : "v"(gptr) : "memory");
        const int s1 = (sj + 1 == NS) ? 0 : sj + 1, s2 = (s1 + 1 == NS) ? 0 : s1 + 1;
        LOADF(A1, B1, s1);
        MMA(A0, B0, 0, 3);
        SCHEDULE;
        { const int kn = (j + NS < NKB) ? j + NS : 0; STAGE(kn, sj); }
        MMA(A0, B0, 3, 4);
        if (full5) MMA(A0, B0, 4, 5);
        WAIT_STAGES(NS - 2);
        asm volatile("s_waitcnt lgkmcnt(0)" ::: "memory");
        BAR;
        LOADF(A0, B0, s2);
        MMA(A1, B1, 0, 3);
        SCHEDULE;
        { const int kn = (j + 1 + NS < NKB) ? j + 1 + NS : 0; STAGE(kn, s1); }
        MMA(A1, B1, 3, 4);
        if (full5) MMA(A1, B1, 4, 5);
        if (j == 8 * EC - 2) {
            const float* xs = (const float*)(myws + OFF_XS) + par * GM;
#pragma unroll
            for (int m = 0; m < 5; ++m) {
                const int f = fbase + (m < fcnt ? m : 0);
                const float full = __hip_atomic_load(xs + (rg0 + f) * 16 + fr, __ATOMIC_RELAXED, __HIP_MEMORY_SCOPE_SYSTEM);
                const float ratio = rsc[m] / full;
                rsc[m] = full;
#pragma unroll
                for (int n = 0; n < 4; ++n) acc[m][n] *= ratio;
            }
        }
        WAIT_STAGES(NS - 2);
        sj = s2;
        if (polling) {
            WAIT_VN(8);
            asm volatile("" : "+v"(pv) :: "memory");
            const unsigned long long bad_ = __ballot(gact && (int)(pv - target) < 0);
            const unsigned both = (unsigned)bad_ | (unsigned)(bad_ >> 32);
            const int r2 = both ? slab_first(__builtin_ctz(both)) : NKB;
            rdy = r2 > rdy ? r2 : rdy;
            constexpr int GA = (NS & 1) ? NS + 3 : NS + 2;
            if (j + GA < NKB && slab_start(j + GA)) {
                GATE(j + GA);
            }
        }
        asm volatile("s_waitcnt lgkmcnt(0)" ::: "memory");
        BAR;
    }
    WAIT_V(0);
    BAR;
    const bool bad = __hip_atomic_load(tmo, __ATOMIC_RELAXED, __HIP_MEMORY_SCOPE_AGENT) == call;
    const float qnan = __builtin_nanf("");
    LAS char* eb = lds + wid * 8704;
#pragma unroll
    for (int m = 0; m < 5; ++m) {
        const int f = fbase + m;
        if (m < fcnt) {
            LAS char* b = eb + (m & 1) * 4352;
#pragma unroll
            for (int n = 0; n < 4; ++n) {
                f32x4 o = (acc[m][n] * rsc[m]) * cs[n];
#pragma unroll
                for (int jj = 0; jj < 4; ++jj) o[jj] = bad ? qnan : (o[jj] < 0.f ? 0.f : o[jj]);
                *(LAS f32x4*)(b + fr * 272 + n * 64 + fq * 16) = o;
            }
            asm volatile("s_waitcnt lgkmcnt(0)" ::: "memory");
            float* op = p.out + (size_t)((rg0 + f) * 16 + fq) * NL + pn * 128 + wc * 64 + fr * 4;
#pragma unroll
            for (int i = 0; i < 4; ++i) {
                const f32x4 o = *(const LAS f32x4*)(b + (i * 4 + fq) * 272 + fr * 16);
                __builtin_nontemporal_store(o, (f32x4*)(op + (size_t)(i * 4) * NL));
            }
        }
    }
#undef STAGE
#undef BAR
#undef LOADF
#undef MMA
#undef GATE
#undef SCHEDULE
#undef WAIT_STAGES
}

__global__ void __launch_bounds__(512, 2) ag_gemm_kernel(Params p) {
    extern __shared__ __attribute__((aligned(16))) char smem[];
    LAS char* lds = (LAS char*)smem;
    const int b = blockIdx.x;
    if (b < PUSH_BLOCKS) role_push(p, lds, b, false);
    else {
        const int g = b - PUSH_BLOCKS;
        unsigned call;
        if (g >= WCONV_BLOCKS && g < WCONV_BLOCKS + RG_PER) call = role_push(p, lds, g - WCONV_BLOCKS, true);
        else {
            call = draw_call((unsigned*)(p.ws[p.rank] + OFF_CTR), (LAS unsigned*)(lds + CTL));
            if (g < WCONV_BLOCKS) role_wconv(p, lds, g, call);
        }
        role_gemm(p, lds, g, call);
    }
}

}

extern "C" void kernel_launch(int rank, int nranks, ncclComm_t comm,
                              hipStream_t stream,
                              void* const* d_in, const int* in_sizes, int n_in,
                              void* d_out, int out_size,
                              const int* in_dtypes, int out_dtype,
                              void* const* d_out_all,
                              void* const* d_ws_all, size_t ws_bytes) {
    static bool init = false;
    if (nranks != NR || ws_bytes < WS_NEED) { fprintf(stderr, "unexpected configuration\n"); return; }
    if (!init) {
        HIP_CHECK(hipFuncSetAttribute((const void*)ag_gemm_kernel, hipFuncAttributeMaxDynamicSharedMemorySize, LDS_BYTES));
        init = true;
    }
    Params p;
    p.x = (const float*)d_in[0];
    p.w = (const float*)d_in[1];
    p.out = (float*)d_out;
    for (int i = 0; i < NR; ++i) p.ws[i] = (char*)d_ws_all[i];
    p.rank = rank;
    ag_gemm_kernel<<<GEMM_BLOCKS + PUSH_BLOCKS, 512, LDS_BYTES, stream>>>(p);
    HIP_CHECK(hipGetLastError());
}
```

```cpp
#include <hip/hip_runtime.h>
#include <rccl/rccl.h>
#include <cstdio>
#include <cstdint>

#define HIP_CHECK(expr) do { hipError_t _e = (expr); if (_e != hipSuccess) { \
    fprintf(stderr, "HIP error %s at %s:%d\n", hipGetErrorString(_e), __FILE__, __LINE__); \
    return; } } while (0)

namespace {

typedef _Float16 h16;
typedef _Float16 h16x8 __attribute__((ext_vector_type(8)));
typedef float f32x4 __attribute__((ext_vector_type(4)));
typedef unsigned u32x4 __attribute__((ext_vector_type(4)));
#define LAS __attribute__((address_space(3)))

constexpr int EC = 1;
constexpr int EWIN = 5;
constexpr int WIN = 5;
constexpr int WIN2 = 3;
constexpr int KSW = 32;
constexpr unsigned long long PACE_T = 60, PACE_0 = 300;

constexpr int NR = 8;
constexpr int GM = 4096;
constexpr int GK = 4096;
constexpr int NL = 2048;
constexpr int MS = GM / NR;
constexpr int NKB = GK / 32;
constexpr int NRG = GM / 16;
constexpr int NCG = NL / 16;
constexpr int RG_PER = MS / 16;

constexpr int UNITS = RG_PER;
__host__ __device__ constexpr bool slab_start(int kb) { return kb < 64 ? (kb & 7) == 0 : (kb < 112 ? (kb & 3) == 0 : (kb & 1) == 0); }
constexpr int NSLAB = 28;
__host__ __device__ constexpr int slab_first(int s) { return s < 8 ? s * 8 : (s < 20 ? 64 + (s - 8) * 4 : 112 + (s - 20) * 2); }
__host__ __device__ constexpr int slab_idx(int kb) { return kb < 64 ? kb >> 3 : (kb < 112 ? 8 + ((kb - 64) >> 2) : 20 + ((kb - 112) >> 1)); }

constexpr int PUSH_BLOCKS = RG_PER;
constexpr int WCONV_BLOCKS = NKB;
constexpr int GEMM_BLOCKS = 224;

constexpr size_t OFF_CTR = 0;
constexpr size_t OFF_TMO = 1024;
constexpr size_t OFF_WC2 = 96 << 10;
constexpr size_t OFF_PMAX = 256 << 10;
constexpr size_t OFF_ARR = 4096;
constexpr size_t OFF_XS  = 128 << 10;
constexpr size_t OFF_XS0 = 160 << 10;
constexpr size_t OFF_WSC = 192 << 10;
constexpr size_t OFF_XH  = 1 << 20;
constexpr size_t XH_BYTES = (size_t)GM * GK * 2;
constexpr size_t OFF_WT  = OFF_XH + 2 * XH_BYTES;
constexpr size_t WS_NEED = OFF_WT + (size_t)NL * GK * 2;

constexpr unsigned long long DEADLINE_TICKS = 50000000ull;

constexpr int NS = 4;
constexpr int STG = 28 * 1024;
constexpr int CTL = (NS * STG > 131072) ? NS * STG : 131072;
constexpr int LDS_BYTES = CTL + 4096;

struct Params {
    const float* x;
    const float* w;
    float* out;
    char* ws[NR];
    int rank;
};

__device__ __forceinline__ unsigned draw_call(unsigned* ctr, LAS unsigned* slot) {
    if (threadIdx.x == 0) {
        unsigned t = __hip_atomic_fetch_add(ctr, 1u, __ATOMIC_RELAXED, __HIP_MEMORY_SCOPE_AGENT);
        *slot = t / gridDim.x + 1u;
    }
    __syncthreads();
    unsigned c = __builtin_amdgcn_readfirstlane(*slot);
    __syncthreads();
    return c;
}

__device__ __forceinline__ void scale_from_amax(float amax, float& mult, float& inv) {
    int ef = (int)((__float_as_uint(amax) >> 23) & 0xffu);
    int e = ef - 127 - 14;
    if (ef == 0xff || amax == 0.f) e = 0;
    e = e < -100 ? -100 : (e > 113 ? 113 : e);
    mult = __uint_as_float((unsigned)(127 - e) << 23);
    inv  = __uint_as_float((unsigned)(127 + e) << 23);
}

__device__ __forceinline__ float amax4(f32x4 v) {
    return fmaxf(fmaxf(fabsf(v[0]), fabsf(v[1])), fmaxf(fabsf(v[2]), fabsf(v[3])));
}

#define WAIT_V(n) asm volatile("s_waitcnt vmcnt(" #n ")" ::: "memory")
#define WAIT_VN(n) asm volatile("s_waitcnt vmcnt(%0)" :: "n"(n) : "memory")

__device__ __forceinline__ unsigned role_push(const Params& p, LAS char* lds, int rg, int mode) {
    const bool local = mode == 1, early = mode == 2;
    typedef _Float16 h16x4 __attribute__((ext_vector_type(4)));
    const int tid = threadIdx.x, wid = __builtin_amdgcn_readfirstlane(tid >> 6), lane = tid & 63;
    const unsigned long long tstart = __builtin_amdgcn_s_memrealtime();
    unsigned ticket = 0;
    if (tid == 0) ticket = __hip_atomic_fetch_add((unsigned*)(p.ws[p.rank] + OFF_CTR), 1u, __ATOMIC_RELAXED, __HIP_MEMORY_SCOPE_AGENT);
    const float* xb = p.x + (size_t)rg * 16 * GK;
    LAS float* rinv = (LAS float*)(lds + CTL + 64);
    f32x4 v[2][16];
#pragma unroll
    for (int c = 0; c < 16; ++c)
#pragma unroll
        for (int r = 0; r < 2; ++r) v[r][c] = *(const f32x4*)(xb + (size_t)(wid * 2 + r) * GK + lane * 4 + ((early && c >= EC) ? 0 : c * 256));
    const int kl = lane * 4;
    const int loff = (kl >> 5) * 1024 + (((kl >> 3) & 3) * 16) * 16 + (kl & 4) * 2;
#pragma unroll
    for (int r = 0; r < 2; ++r) {
        const int row = wid * 2 + r;
        float am = 0.f;
#pragma unroll
        for (int c = 0; c < EC; ++c) am = fmaxf(am, amax4(v[r][c]));
#pragma unroll
        for (int o = 32; o >= 1; o >>= 1) am = fmaxf(am, __shfl_xor(am, o));
        float mult, inv;
        scale_from_amax(am, mult, inv);
        if (lane == 0) rinv[16 + row] = inv;
#pragma unroll
        for (int c = 0; c < EC; ++c) {
            h16x4 h;
#pragma unroll
            for (int j = 0; j < 4; ++j) h[j] = (h16)(v[r][c][j] * mult);
            *(LAS h16x4*)(lds + c * 8192 + loff + row * 16) = h;
        }
    }
    if (tid == 0) *(LAS unsigned*)(lds + CTL) = ticket / gridDim.x + 1u;
    __syncthreads();
    const unsigned call = __builtin_amdgcn_readfirstlane(*(LAS unsigned*)(lds + CTL));
    const int par = call & 1;
    const int grg = p.rank * RG_PER + rg;
    const bool sender = wid < 7;
    char* dws = p.ws[(p.rank + 1 + wid) & 7];
    __amdgpu_buffer_rsrc_t rs = __builtin_amdgcn_make_buffer_rsrc(
        (void*)(dws + OFF_XH + (size_t)par * XH_BYTES), 0, (int)XH_BYTES, 0x00020000);
    unsigned* arr = (unsigned*)(dws + OFF_ARR + (size_t)p.rank * 256);
    if (early) {
        if (!sender) return call;
        if (lane < 16) __hip_atomic_store((float*)(dws + OFF_XS0) + par * GM + grg * 16 + lane, rinv[16 + lane], __ATOMIC_RELAXED, __HIP_MEMORY_SCOPE_SYSTEM);
#pragma unroll
        for (int kb = 0; kb < 8 * EC; ++kb) {
            u32x4 dat = *(const LAS u32x4*)(lds + kb * 1024 + lane * 16);
            if (kb >= EWIN) WAIT_VN(EWIN - 1);
            __builtin_amdgcn_raw_buffer_store_b128(dat, rs, (kb * NRG + grg) * 1024 + lane * 16, 0, 17  );
        }
        WAIT_V(0);
        if (lane < EC) __hip_atomic_fetch_add(arr + lane, 1u, __ATOMIC_RELAXED, __HIP_MEMORY_SCOPE_SYSTEM);
        return call;
    }
#pragma unroll
    for (int r = 0; r < 2; ++r) {
        const int row = wid * 2 + r;
        float am = 0.f;
#pragma unroll
        for (int c = 0; c < 16; ++c) am = fmaxf(am, amax4(v[r][c]));
#pragma unroll
        for (int o = 32; o >= 1; o >>= 1) am = fmaxf(am, __shfl_xor(am, o));
        float mult, inv;
        scale_from_amax(am, mult, inv);
        if (lane == 0) rinv[row] = inv;
#pragma unroll
        for (int c = EC; c < 16; ++c) {
            h16x4 h;
#pragma unroll
            for (int j = 0; j < 4; ++j) h[j] = (h16)(v[r][c][j] * mult);
            *(LAS h16x4*)(lds + c * 8192 + loff + row * 16) = h;
        }
    }
    __syncthreads();
    if (local) {
        dws = p.ws[p.rank];
        rs = __builtin_amdgcn_make_buffer_rsrc((void*)(dws + OFF_XH + (size_t)par * XH_BYTES), 0, (int)XH_BYTES, 0x00020000);
        if (tid < 16) __hip_atomic_store((float*)(dws + OFF_XS) + par * GM + grg * 16 + tid, rinv[tid], __ATOMIC_RELAXED, __HIP_MEMORY_SCOPE_AGENT);
        else if (tid < 32) __hip_atomic_store((float*)(dws + OFF_XS0) + par * GM + grg * 16 + (tid - 16), rinv[tid], __ATOMIC_RELAXED, __HIP_MEMORY_SCOPE_AGENT);
#pragma unroll
        for (int i = 0; i < 16; ++i) {
            const int kb = i * 8 + wid;
            u32x4 dat = *(const LAS u32x4*)(lds + kb * 1024 + lane * 16);
            __builtin_amdgcn_raw_buffer_store_b128(dat, rs, (kb * NRG + grg) * 1024 + lane * 16, 0, 16  );
        }
        WAIT_V(0);
        __syncthreads();
        if (tid == 0) __hip_atomic_fetch_add((unsigned*)(dws + OFF_ARR + (size_t)p.rank * 256), 1u, __ATOMIC_RELAXED, __HIP_MEMORY_SCOPE_AGENT);
        return call;
    }
    if (!sender) return call;
    if (lane < 16) __hip_atomic_store((float*)(dws + OFF_XS) + par * GM + grg * 16 + lane, rinv[lane], __ATOMIC_RELAXED, __HIP_MEMORY_SCOPE_SYSTEM);
    int sig = 8 * EC;
#define PACE(kb) do { const unsigned long long due = tstart + PACE_0 + (unsigned long long)(kb) * PACE_T; \
        if (__builtin_amdgcn_s_memrealtime() < due) { \
            WAIT_V(0); \
            for (; sig < (kb); ++sig) \
                if (slab_start(sig + 1) && lane == 0) __hip_atomic_fetch_add(arr + slab_idx(sig), 1u, __ATOMIC_RELAXED, __HIP_MEMORY_SCOPE_SYSTEM); \
            while (__builtin_amdgcn_s_memrealtime() < due) __builtin_amdgcn_s_sleep(1); } } while (0)
#define PUSH_RANGE(k0, k1, W) \
    for (int b4 = (k0); b4 < (k1); b4 += 4) { \
        PACE(b4); \
        _Pragma("unroll") for (int i = 0; i < 4; ++i) { \
            const int kb = b4 + i; \
            u32x4 dat = *(const LAS u32x4*)(lds + kb * 1024 + lane * 16); \
            WAIT_VN((W) - 1);                               \
            for (; sig <= kb - (W); ++sig) \
                if (slab_start(sig + 1) && lane == 0) __hip_atomic_fetch_add(arr + slab_idx(sig), 1u, __ATOMIC_RELAXED, __HIP_MEMORY_SCOPE_SYSTEM); \
            __builtin_amdgcn_raw_buffer_store_b128(dat, rs, (kb * NRG + grg) * 1024 + lane * 16, 0, 17  ); \
        } \
    }
    PUSH_RANGE(8 * EC, KSW, WIN)
    PUSH_RANGE(KSW, NKB, WIN2)
#undef PUSH_RANGE
#undef PACE
    WAIT_V(0);
    for (; sig < NKB; ++sig)
        if ((sig + 1 == NKB || slab_start(sig + 1)) && lane == 0) __hip_atomic_fetch_add(arr + slab_idx(sig), 1u, __ATOMIC_RELAXED, __HIP_MEMORY_SCOPE_SYSTEM);
    return call;
}

__device__ __forceinline__ void poll(const unsigned* ctr, unsigned target, unsigned* tmo, unsigned call, int lane);
__device__ __forceinline__ void role_wconv(const Params& p, LAS char* lds, int g, unsigned call) {
    const int tid = threadIdx.x, wid = __builtin_amdgcn_readfirstlane(tid >> 6), lane = tid & 63;
    char* myws = p.ws[p.rank];
    const int sl = g >> 5, cq = lane & 15, col0 = (g & 31) * 64 + cq * 4;
    const int kbl = wid * 4 + (lane >> 4), kb = sl * 32 + kbl;
    const float* src = p.w + (size_t)(kb * 32) * NL + col0;
    f32x4 v[32];
#pragma unroll
    for (int r = 0; r < 32; ++r) v[r] = *(const f32x4*)(src + (size_t)r * NL);
    f32x4 am = {0.f, 0.f, 0.f, 0.f};
#pragma unroll
    for (int r = 0; r < 32; ++r)
#pragma unroll
        for (int j = 0; j < 4; ++j) am[j] = fmaxf(am[j], fabsf(v[r][j]));
#pragma unroll
    for (int j = 0; j < 4; ++j) { am[j] = fmaxf(am[j], __shfl_xor(am[j], 16)); am[j] = fmaxf(am[j], __shfl_xor(am[j], 32)); }
    LAS f32x4* red = (LAS f32x4*)(lds + CTL + 1024);
    if (lane < 16) red[wid * 16 + lane] = am;
    __syncthreads();
#pragma unroll
    for (int w = 0; w < 8; ++w) { f32x4 o = red[w * 16 + cq];
#pragma unroll
        for (int j = 0; j < 4; ++j) am[j] = fmaxf(am[j], o[j]); }
    float* pmax = (float*)(myws + OFF_PMAX);
    if (wid == 0) {
        if (lane < 16) {
#pragma unroll
            for (int j = 0; j < 4; ++j) __hip_atomic_store(pmax + sl * NL + col0 + j, am[j], __ATOMIC_RELAXED, __HIP_MEMORY_SCOPE_AGENT);
        }
        WAIT_V(0);
        unsigned* c2 = (unsigned*)(myws + OFF_WC2 + (g & 31) * 128);
        if (lane == 0) __hip_atomic_fetch_add(c2, 1u, __ATOMIC_RELAXED, __HIP_MEMORY_SCOPE_AGENT);
        poll(c2, call * 4u, (unsigned*)(myws + OFF_TMO), call, lane);
    }
    __syncthreads();
#pragma unroll
    for (int q4 = 0; q4 < 4; ++q4)
#pragma unroll
        for (int j = 0; j < 4; ++j) am[j] = fmaxf(am[j], __hip_atomic_load(pmax + q4 * NL + col0 + j, __ATOMIC_RELAXED, __HIP_MEMORY_SCOPE_AGENT));
    float mult[4], inv[4];
#pragma unroll
    for (int j = 0; j < 4; ++j) scale_from_amax(am[j], mult[j], inv[j]);
    if (sl == 0 && tid < 16) {
#pragma unroll
        for (int j = 0; j < 4; ++j) __hip_atomic_store((float*)(myws + OFF_WSC) + col0 + j, inv[j], __ATOMIC_RELAXED, __HIP_MEMORY_SCOPE_AGENT);
    }
    LAS char* fb = lds + wid * 16384;
    const int cgl = cq >> 2, c0 = (cq & 3) * 4;
#pragma unroll
    for (int kq = 0; kq < 4; ++kq)
#pragma unroll
        for (int j = 0; j < 4; ++j) {
            h16x8 h;
#pragma unroll
            for (int r = 0; r < 8; ++r) h[r] = (h16)(v[kq * 8 + r][j] * mult[j]);
            *(LAS h16x8*)(fb + ((lane >> 4) * 4 + cgl) * 1024 + (kq * 16 + c0 + j) * 16) = h;
        }
    asm volatile("s_waitcnt lgkmcnt(0)" ::: "memory");
    __amdgpu_buffer_rsrc_t rs = __builtin_amdgcn_make_buffer_rsrc((void*)(myws + OFF_WT), 0, NL * GK * 2, 0x00020000);
#pragma unroll
    for (int f = 0; f < 16; ++f) {
        const u32x4 dat = *(const LAS u32x4*)(fb + f * 1024 + lane * 16);
        const int kbf = sl * 32 + wid * 4 + (f >> 2), cg = (g & 31) * 4 + (f & 3);
        __builtin_amdgcn_raw_buffer_store_b128(dat, rs, (kbf * NCG + cg) * 1024 + lane * 16, 0, 16  );
    }
    WAIT_V(0);
    __syncthreads();
    if (tid == 0) __hip_atomic_fetch_add((unsigned*)(myws + OFF_WC2 + (g & 31) * 128 + 64), 1u, __ATOMIC_RELAXED, __HIP_MEMORY_SCOPE_AGENT);
}

__device__ __forceinline__ void poll(const unsigned* ctr, unsigned target, unsigned* tmo, unsigned call, int lane) {
    const unsigned long long t0 = __builtin_amdgcn_s_memrealtime();
    while ((int)((unsigned)__builtin_amdgcn_readfirstlane(__hip_atomic_load(ctr, __ATOMIC_RELAXED, __HIP_MEMORY_SCOPE_SYSTEM)) - target) < 0) {
        if ((unsigned)__builtin_amdgcn_readfirstlane(__hip_atomic_load(tmo, __ATOMIC_RELAXED, __HIP_MEMORY_SCOPE_AGENT)) == call
            || __builtin_amdgcn_s_memrealtime() - t0 > DEADLINE_TICKS) {
            if (lane == 0) __hip_atomic_store(tmo, call, __ATOMIC_RELAXED, __HIP_MEMORY_SCOPE_AGENT);
            break;
        }
        __builtin_amdgcn_s_sleep(2);
    }
}

__device__ __forceinline__ void role_gemm(const Params& p, LAS char* lds, int bid, unsigned call) {
    const int tid = threadIdx.x, wid = __builtin_amdgcn_readfirstlane(tid >> 6), lane = tid & 63;
    const int wr = wid >> 1, wc = wid & 1;
    char* myws = p.ws[p.rank];
    const int par = call & 1;
    unsigned* tmo = (unsigned*)(myws + OFF_TMO);

    const int xcd = bid & 7, sl = bid >> 3;
    const int band = 7 * (xcd >> 2) + (sl % 7), pn = 4 * (xcd & 3) + (sl / 7);
    const int rg0 = band * 18 + (band < 4 ? band : 4);
    const int nrg = band < 4 ? 19 : 18;
    const int srcA = rg0 / RG_PER, srcB = (rg0 + nrg - 1) / RG_PER;
    const bool needA = srcA != p.rank, needB = (srcB != srcA) && (srcB != p.rank);
    const bool needL = (srcA == p.rank) || (srcB == p.rank);
    const unsigned* arrA = (const unsigned*)(myws + OFF_ARR + (size_t)srcA * 256);
    const unsigned* arrB = (const unsigned*)(myws + OFF_ARR + (size_t)srcB * 256);
    const unsigned target = call * UNITS;

    const char* gsrc[4];
    int fidx[4];
#pragma unroll
    for (int i = 0; i < 4; ++i) {
        const int f = (i < 3) ? wid * 3 + i : 24 + wid;
        fidx[i] = f;
        const int ra = rg0 + (f < nrg ? f : nrg - 1);
        gsrc[i] = (f < 20) ? myws + OFF_XH + (size_t)par * XH_BYTES + (size_t)ra * 1024 + lane * 16
                           : myws + OFF_WT + (size_t)(pn * 8 + ((f - 20) & 7)) * 1024 + lane * 16;
    }
#define STAGE(kb, slot) do { _Pragma("unroll") for (int _i = 0; _i < 4; ++_i) { if (_i < 3 || wid < 4) \
        __builtin_amdgcn_global_load_lds((const unsigned*)(gsrc[_i] + (size_t)(kb) * ((fidx[_i] < 20) ? NRG * 1024 : NCG * 1024)), \
            (LAS unsigned*)(lds + (slot) * STG + fidx[_i] * 1024), 16, 0, 0); } } while (0)
#define BAR __builtin_amdgcn_s_barrier()
#define WAIT_STAGES(n) do { if (wid < 4) WAIT_VN((n) * 4); else WAIT_VN((n) * 3); } while (0)
    f32x4 acc[5][4];
#pragma unroll
    for (int m = 0; m < 5; ++m)
#pragma unroll
        for (int n = 0; n < 4; ++n) acc[m][n] = (f32x4){0.f, 0.f, 0.f, 0.f};
    h16x8 A0[5], B0[4], A1[5], B1[4];
    const int fbase = (wr < 3) ? wr * 5 : nrg - 4;
    const int fcnt = (wr < 2) ? 5 : ((wr == 2) ? nrg - 14 : 4);
    const bool full5 = fcnt == 5;
    const int aoff = fbase * 1024 + lane * 16, boff = (20 + wc * 4) * 1024 + lane * 16;
#define LOADF(Af, Bf, slot) do { const LAS char* _b = lds + (slot) * STG; \
        _Pragma("unroll") for (int n = 0; n < 4; ++n) Bf[n] = *(const LAS h16x8*)(_b + boff + n * 1024); \
        _Pragma("unroll") for (int m = 0; m < 5; ++m) Af[m] = *(const LAS h16x8*)(_b + aoff + m * 1024); } while (0)
#define MMA(Af, Bf, m0, m1) do { _Pragma("unroll") for (int m = m0; m < m1; ++m) _Pragma("unroll") for (int n = 0; n < 4; ++n) \
        acc[m][n] = __builtin_amdgcn_mfma_f32_16x16x32_f16(Bf[n], Af[m], acc[m][n], 0, 0, 0); } while (0)
#define SCHEDULE do { _Pragma("unroll") for (int _q = 0; _q < 9; ++_q) { __builtin_amdgcn_sched_group_barrier(0x8, 1, 0); __builtin_amdgcn_sched_group_barrier(0x100, 1, 0); } \
        __builtin_amdgcn_sched_group_barrier(0x8, 3, 0); } while (0)
    int rdy = (needA || needB) ? 0 : NKB;
    const int gl = lane & 31;
    const bool gact = gl < NSLAB && ((lane < 32) ? needA : needB);
    const unsigned* gptr = ((lane < 32) ? arrA : arrB) + gl;
#define GATE(kb) do { if ((kb) >= rdy) { const unsigned long long t0 = __builtin_amdgcn_s_memrealtime(); \
        for (;;) { const unsigned v = __hip_atomic_load(gptr, __ATOMIC_RELAXED, __HIP_MEMORY_SCOPE_SYSTEM); \
            const unsigned long long bad_ = __ballot(gact && (int)(v - target) < 0); \
            const unsigned both = (unsigned)bad_ | (unsigned)(bad_ >> 32); \
            rdy = both ? slab_first(__builtin_ctz(both)) : NKB; \
            if ((kb) < rdy) break; \
            if ((unsigned)__builtin_amdgcn_readfirstlane(__hip_atomic_load(tmo, __ATOMIC_RELAXED, __HIP_MEMORY_SCOPE_AGENT)) == call \
                || __builtin_amdgcn_s_memrealtime() - t0 > DEADLINE_TICKS) { \
                if (lane == 0) __hip_atomic_store(tmo, call, __ATOMIC_RELAXED, __HIP_MEMORY_SCOPE_AGENT); rdy = NKB; break; } \
            __builtin_amdgcn_s_sleep(1); } } } while (0)

    if (wid == 0) {
        poll((const unsigned*)(myws + OFF_WC2 + (2 * pn) * 128 + 64), call * 4u, tmo, call, lane);
        poll((const unsigned*)(myws + OFF_WC2 + (2 * pn + 1) * 128 + 64), call * 4u, tmo, call, lane);
        if (needL) poll((const unsigned*)(myws + OFF_ARR + (size_t)p.rank * 256), target, tmo, call, lane);
        GATE(0);
        __builtin_amdgcn_fence(__ATOMIC_ACQUIRE, "");
        WAIT_V(0);
    }
    asm volatile("" ::: "memory"); BAR; asm volatile("" ::: "memory");
    const int fr = lane & 15, fq = lane >> 4;
    f32x4 cs[4];
    float rsc[5];
    {
        const float* xs = (const float*)(myws + OFF_XS0) + par * GM;
        const float* wsc = (const float*)(myws + OFF_WSC) + pn * 128 + wc * 64 + 4 * fq;
#pragma unroll
        for (int n = 0; n < 4; ++n)
#pragma unroll
            for (int jj = 0; jj < 4; ++jj) cs[n][jj] = __hip_atomic_load(wsc + n * 16 + jj, __ATOMIC_RELAXED, __HIP_MEMORY_SCOPE_AGENT);
#pragma unroll
        for (int m = 0; m < 5; ++m) {
            const int f = fbase + (m < fcnt ? m : 0);
            rsc[m] = __hip_atomic_load(xs + (rg0 + f) * 16 + fr, __ATOMIC_RELAXED, __HIP_MEMORY_SCOPE_SYSTEM);
        }
        WAIT_V(0);
    }
#pragma unroll
    for (int i = 0; i < NS; ++i) STAGE(i, i);
    WAIT_STAGES(NS - 1); BAR;
    LOADF(A0, B0, 0);
    WAIT_STAGES(NS - 2);
    asm volatile("s_waitcnt lgkmcnt(0)" ::: "memory");
    BAR;
    int sj = 0;
    for (int j = 0; j < NKB; j += 2) {
        unsigned pv = 0;
        const bool polling = (wid == 0) && (rdy < NKB) && (rdy <= j + 16);
        if (polling) asm volatile("global_load_dword %0, %1, off sc0 sc1" : "=v"(pv) : "v"(gptr) : "memory");
        const int s1 = (sj + 1 == NS) ? 0 : sj + 1, s2 = (s1 + 1 == NS) ? 0 : s1 + 1;
        LOADF(A1, B1, s1);
        MMA(A0, B0, 0, 3);
        SCHEDULE;
        { const int kn = (j + NS < NKB) ? j + NS : 0; STAGE(kn, sj); }
        MMA(A0, B0, 3, 4);
        if (full5) MMA(A0, B0, 4, 5);
        WAIT_STAGES(NS - 2);
        asm volatile("s_waitcnt lgkmcnt(0)" ::: "memory");
        BAR;
        LOADF(A0, B0, s2);
        MMA(A1, B1, 0, 3);
        SCHEDULE;
        { const int kn = (j + 1 + NS < NKB) ? j + 1 + NS : 0; STAGE(kn, s1); }
        MMA(A1, B1, 3, 4);
        if (full5) MMA(A1, B1, 4, 5);
        if (j == 8 * EC - 2) {
            const float* xs = (const float*)(myws + OFF_XS) + par * GM;
#pragma unroll
            for (int m = 0; m < 5; ++m) {
                const int f = fbase + (m < fcnt ? m : 0);
                const float full = __hip_atomic_load(xs + (rg0 + f) * 16 + fr, __ATOMIC_RELAXED, __HIP_MEMORY_SCOPE_SYSTEM);
                const float ratio = rsc[m] / full;
                rsc[m] = full;
#pragma unroll
                for (int n = 0; n < 4; ++n) acc[m][n] *= ratio;
            }
        }
        WAIT_STAGES(NS - 2);
        sj = s2;
        if (polling) {
            WAIT_VN(8);
            asm volatile("" : "+v"(pv) :: "memory");
            const unsigned long long bad_ = __ballot(gact && (int)(pv - target) < 0);
            const unsigned both = (unsigned)bad_ | (unsigned)(bad_ >> 32);
            const int r2 = both ? slab_first(__builtin_ctz(both)) : NKB;
            rdy = r2 > rdy ? r2 : rdy;
            constexpr int GA = (NS & 1) ? NS + 3 : NS + 2;
            if (j + GA < NKB && slab_start(j + GA)) {
                GATE(j + GA);
            }
        }
        asm volatile("s_waitcnt lgkmcnt(0)" ::: "memory");
        BAR;
    }
    WAIT_V(0);
    BAR;
    const bool bad = __hip_atomic_load(tmo, __ATOMIC_RELAXED, __HIP_MEMORY_SCOPE_AGENT) == call;
    const float qnan = __builtin_nanf("");
    LAS char* eb = lds + wid * 8704;
#pragma unroll
    for (int m = 0; m < 5; ++m) {
        const int f = fbase + m;
        if (m < fcnt) {
            LAS char* b = eb + (m & 1) * 4352;
#pragma unroll
            for (int n = 0; n < 4; ++n) {
                f32x4 o = (acc[m][n] * rsc[m]) * cs[n];
#pragma unroll
                for (int jj = 0; jj < 4; ++jj) o[jj] = bad ? qnan : (o[jj] < 0.f ? 0.f : o[jj]);
                *(LAS f32x4*)(b + fr * 272 + n * 64 + fq * 16) = o;
            }
            asm volatile("s_waitcnt lgkmcnt(0)" ::: "memory");
            float* op = p.out + (size_t)((rg0 + f) * 16 + fq) * NL + pn * 128 + wc * 64 + fr * 4;
#pragma unroll
            for (int i = 0; i < 4; ++i) {
                const f32x4 o = *(const LAS f32x4*)(b + (i * 4 + fq) * 272 + fr * 16);
                __builtin_nontemporal_store(o, (f32x4*)(op + (size_t)(i * 4) * NL));
            }
        }
    }
#undef STAGE
#undef BAR
#undef LOADF
#undef MMA
#undef GATE
#undef SCHEDULE
#undef WAIT_STAGES
}

__global__ void __launch_bounds__(512, 2) ag_gemm_kernel(Params p) {
    extern __shared__ __attribute__((aligned(16))) char smem[];
    LAS char* lds = (LAS char*)smem;
    const int b = blockIdx.x;
    if (b < PUSH_BLOCKS) role_push(p, lds, b, 0);
    else {
        const int g = b - PUSH_BLOCKS;
        unsigned call;
        if (g >= WCONV_BLOCKS && g < WCONV_BLOCKS + 2 * RG_PER) call = role_push(p, lds, (g - WCONV_BLOCKS) & (RG_PER - 1), 1 + ((g - WCONV_BLOCKS) >> 5));
        else {
            call = draw_call((unsigned*)(p.ws[p.rank] + OFF_CTR), (LAS unsigned*)(lds + CTL));
            if (g < WCONV_BLOCKS) role_wconv(p, lds, g, call);
        }
        role_gemm(p, lds, g, call);
    }
}

}

extern "C" void kernel_launch(int rank, int nranks, ncclComm_t comm,
                              hipStream_t stream,
                              void* const* d_in, const int* in_sizes, int n_in,
                              void* d_out, int out_size,
                              const int* in_dtypes, int out_dtype,
                              void* const* d_out_all,
                              void* const* d_ws_all, size_t ws_bytes) {
    static bool init = false;
    if (nranks != NR || ws_bytes < WS_NEED) { fprintf(stderr, "unexpected configuration\n"); return; }
    if (!init) {
        HIP_CHECK(hipFuncSetAttribute((const void*)ag_gemm_kernel, hipFuncAttributeMaxDynamicSharedMemorySize, LDS_BYTES));
        init = true;
    }
    Params p;
    p.x = (const float*)d_in[0];
    p.w = (const float*)d_in[1];
    p.out = (float*)d_out;
    for (int i = 0; i < NR; ++i) p.ws[i] = (char*)d_ws_all[i];
    p.rank = rank;
    ag_gemm_kernel<<<GEMM_BLOCKS + PUSH_BLOCKS, 512, LDS_BYTES, stream>>>(p);
    HIP_CHECK(hipGetLastError());
}
```
